```python
import numpy as np

kernel_name = "gpu_gemm_a2a_m4096_k4096_n16384_f32_none"


ABSMAX_RTOL = 0.001

COMPUTE_DTYPE = "float32"
EPILOGUE = "none"
IN_QUANT = None
OUT_QUANT = None
IN_DTYPES = {"x": "float32", "w_mat": "float32"}
WORLD_SIZE = 8

def setup_inputs():

    x = np.random.randn(4096, 4096).astype(np.float32)
    w_mat = (np.random.randn(4096, 16384) / np.sqrt(4096)).astype(np.float32)
    return {"x": x, "w_mat": w_mat}

def _epilogue(y):
    return y

def reference(x, w_mat):
    y = (x @ w_mat).astype(np.float32)
    return _epilogue(y)
```

```cpp
#include <hip/hip_runtime.h>
#include <rccl/rccl.h>
#include <cstdio>

#define HIP_CHECK(expr) do { hipError_t _e = (expr); if (_e != hipSuccess) { \
    fprintf(stderr, "HIP error %s at %s:%d\n", hipGetErrorString(_e), __FILE__, __LINE__); \
    return; } } while (0)

#define LAS __attribute__((address_space(3)))
typedef _Float16 h8 __attribute__((ext_vector_type(8)));
typedef _Float16 h2 __attribute__((ext_vector_type(2)));
typedef float f4 __attribute__((ext_vector_type(4)));
typedef float f2 __attribute__((ext_vector_type(2)));
typedef unsigned u4 __attribute__((ext_vector_type(4)));

namespace {

constexpr int NR = 8;
constexpr int M_SH = 512, KD = 4096, N_SH = 2048, N_TOT = 16384;
constexpr int BM = 128, BN = 256, BK = 32;
constexpr int NST = KD / BK;
constexpr int A_ST = BM * BK * 2;
constexpr int B_ST = BN * BK * 2;
constexpr int ST = A_ST + B_ST;
constexpr int NSLOT = 6;
constexpr int CTRL = NSLOT * ST;
constexpr int LDS_BYTES = CTRL + 64;
constexpr int NBLK = 256;

#define NP 8

constexpr size_t WS_CALL = 0;
constexpr size_t WS_TMO = 32768;
constexpr size_t WS_FLAGA = 65536;
constexpr size_t WS_FLAGW = WS_FLAGA + 8 * 4 * 512;
constexpr size_t WS_DBG = 262144;
constexpr size_t WS_X16 = 1u << 20;
constexpr size_t X16_PAR = (size_t)NR * 4 * NST * A_ST;
constexpr size_t WS_W16 = WS_X16 + 2 * X16_PAR;
constexpr size_t WS_END = WS_W16 + (size_t)8 * NST * B_ST;

constexpr unsigned long long DEADLINE_TICKS = 40000000ull;

struct Params {
  const float* x;
  const float* w;
  float* out;
  char* ws[NR];
  int rank;
  int pad;
};

#define SCHED() __builtin_amdgcn_sched_barrier(0)
#define CFENCE() asm volatile("" ::: "memory")
#define WAIT_V(n) asm volatile("s_waitcnt vmcnt(" #n ")" ::: "memory")
#define WAIT_L(n) asm volatile("s_waitcnt lgkmcnt(" #n ")" ::: "memory")

__device__ __forceinline__ unsigned rfl(unsigned v) { return __builtin_amdgcn_readfirstlane(v); }
__device__ __forceinline__ const char* rfl_ptr(const char* p) {
  unsigned long long v = (unsigned long long)p;
  unsigned lo = rfl((unsigned)v), hi = rfl((unsigned)(v >> 32));
  return (const char*)(((unsigned long long)hi << 32) | lo);
}

__device__ __forceinline__ unsigned cvt2(float a, float b) {
  f2 v = {a, b};
  h2 h = __builtin_convertvector(v, h2);
  return __builtin_bit_cast(unsigned, h);
}

__device__ __forceinline__ void glds8k(const char* gbase, unsigned voff, unsigned lds_dst) {
  unsigned keep;
  const char* g2 = gbase + 4096;
  unsigned d2 = lds_dst + 4096;
  asm volatile(
      "s_mov_b32 %0, m0\n\t"
      "s_mov_b32 m0, %4\n\t"
      "s_nop 0\n\t"
      "global_load_lds_dwordx4 %1, %2 sc0 sc1\n\t"
      "global_load_lds_dwordx4 %1, %2 offset:1024 sc0 sc1\n\t"
      "global_load_lds_dwordx4 %1, %2 offset:2048 sc0 sc1\n\t"
      "global_load_lds_dwordx4 %1, %2 offset:3072 sc0 sc1\n\t"
      "s_mov_b32 m0, %5\n\t"
      "s_nop 0\n\t"
      "global_load_lds_dwordx4 %1, %3 sc0 sc1\n\t"
      "global_load_lds_dwordx4 %1, %3 offset:1024 sc0 sc1\n\t"
      "global_load_lds_dwordx4 %1, %3 offset:2048 sc0 sc1\n\t"
      "global_load_lds_dwordx4 %1, %3 offset:3072 sc0 sc1\n\t"
      "s_mov_b32 m0, %0"
      : "=&s"(keep)
      : "v"(voff), "s"(gbase), "s"(g2), "s"(lds_dst), "s"(d2)
      : "memory");
}

__device__ __forceinline__ void lds_st(unsigned addr, unsigned v) {
  asm volatile("ds_write_b32 %0, %1" ::"v"(addr), "v"(v) : "memory");
}
__device__ __forceinline__ u4 lds_ld4_wait(unsigned addr) {
  u4 r;
  asm volatile("ds_read_b128 %0, %1\n\ts_waitcnt lgkmcnt(0)" : "=&v"(r) : "v"(addr) : "memory");
  return r;
}
__device__ __forceinline__ void flag_store(const char* sbase, unsigned voff, unsigned val) {
  unsigned long long keep;
  asm volatile(
      "s_mov_b64 %0, exec\n\t"
      "s_mov_b64 exec, 1\n\t"
      "global_store_dword %1, %2, %3 sc0 sc1\n\t"
      "s_mov_b64 exec, %0"
      : "=&s"(keep)
      : "v"(voff), "v"(val), "s"(sbase)
      : "memory");
}

#define STAMP(i) do {} while (0)

__global__ __launch_bounds__(512, 2) void fused_kernel(Params p) {
  extern __shared__ __attribute__((aligned(1024))) char smem[];
  const unsigned lds0 = (unsigned)(unsigned long long)(LAS char*)smem;
  const int tid = threadIdx.x;
  const int lane = tid & 63;
  const int wave = (int)rfl((unsigned)(tid >> 6));
  const int bid = blockIdx.x;
  const int xcd = bid & 7, idx = bid >> 3;
  const int rbg = (xcd >> 1) * 8 + (idx >> 2);
  const int cb = (xcd & 1) * 4 + (idx & 3);
  const int rank = p.rank;
  char* const myws = p.ws[rank];

  if (wave == 0) STAMP(0);

  unsigned* const callp = (unsigned*)(myws + WS_CALL + (size_t)bid * 128) + wave;
  unsigned* const tmo = (unsigned*)(myws + WS_TMO);

  if (tid < 16) ((LAS unsigned*)(smem + CTRL))[tid] = 0u;
  WAIT_L(0);
  __builtin_amdgcn_s_barrier();
  CFENCE();

  if (wave < 4) {
    const int img = bid * 4 + wave;
    const int wcb = img & 7, wks = img >> 3;
    const int li = lane & 15, ko = lane >> 4;
    const float* wsrc = p.w + (size_t)(wks * BK + ko * 8) * N_TOT + (size_t)rank * N_SH + wcb * BN + li * 4;
    const unsigned callv = __hip_atomic_load(callp, __ATOMIC_RELAXED, __HIP_MEMORY_SCOPE_AGENT);
    f4 v[4][8];
#pragma unroll
    for (int g = 0; g < 4; ++g)
#pragma unroll
      for (int t = 0; t < 8; ++t) v[g][t] = *(const f4*)(wsrc + (size_t)t * N_TOT + g * 64);
    SCHED();
    const unsigned call = rfl(callv) + 1u;
    {
      const char* dst = rfl_ptr(myws + WS_W16 + ((size_t)wcb * NST + wks) * B_ST);
      const unsigned voff = lane * 16;
#pragma unroll
      for (int g = 0; g < 4; ++g) {
        const char* dg = dst + g * 4096;
#pragma unroll
        for (int j = 0; j < 4; ++j) {
          u4 o;
          o.x = cvt2(v[g][0][j], v[g][1][j]);
          o.y = cvt2(v[g][2][j], v[g][3][j]);
          o.z = cvt2(v[g][4][j], v[g][5][j]);
          o.w = cvt2(v[g][6][j], v[g][7][j]);
          asm volatile("global_store_dwordx4 %0, %1, %2 offset:%3 sc0 sc1\n\ts_nop 1"
                       ::"v"(voff), "v"(o), "s"(dg), "i"(j * 1024) : "memory");
        }
      }
      WAIT_V(0);
      flag_store(rfl_ptr(myws + WS_FLAGW + (size_t)(wcb * NST + wks) * 4), 0u, call);
      if (lane == 0) __hip_atomic_store(callp, call, __ATOMIC_RELAXED, __HIP_MEMORY_SCOPE_AGENT);
    }
    if (wave == 0) STAMP(1);

    const int wm = wave >> 1, wn = wave & 1;
    const unsigned arow = lane & 15, ako = lane >> 4;
    const unsigned offA = lds0 + wm * 4096 + arow * 64 + ((ako ^ ((arow >> 3) << 1)) << 4);
    const unsigned offB = lds0 + A_ST + wn * 8192 + lane * 16;
    const unsigned fullAddr = lds0 + CTRL;
    const unsigned freeAddr = lds0 + CTRL + 16 + wave * 4;

    f4 acc[4][8];
#pragma unroll
    for (int i = 0; i < 4; ++i)
#pragma unroll
      for (int j = 0; j < 8; ++j) acc[i][j] = (f4){0.f, 0.f, 0.f, 0.f};
    h8 a0[4], a1[4], b[8];

#define LDSRD(addr) (*(const LAS h8*)(addr))
    unsigned ready = 0;
    auto wait_ready = [&](unsigned need) {
      while (ready < need) {
        u4 f = lds_ld4_wait(fullAddr);
        unsigned m = f.x < f.y ? f.x : f.y;
        m = m < f.z ? m : f.z;
        ready = rfl(m);
        if (ready < need) __builtin_amdgcn_s_sleep(1);
      }
    };
    wait_ready(1);
    if (wave == 0) STAMP(2);
#pragma unroll
    for (int i = 0; i < 4; ++i) a0[i] = LDSRD(offA + i * 1024);
#pragma unroll
    for (int j = 0; j < 8; ++j) b[j] = LDSRD(offB + j * 1024);

    unsigned slotoff = 0;
#define STAGE(ac, an, s)                                                                     \
    {                                                                                        \
      if (ready < (unsigned)(s) + 2u) wait_ready((unsigned)(s) + 2u);                        \
      slotoff = slotoff + ST == (unsigned)CTRL ? 0u : slotoff + ST;                          \
      const unsigned vA = offA + slotoff, vB = offB + slotoff;                               \
      u4 fw;                                                                                 \
      _Pragma("unroll") for (int j = 0; j < 8; ++j) {                                        \
        SCHED();                                                                             \
        _Pragma("unroll") for (int i = 0; i < 4; ++i)                                        \
          acc[i][j] = __builtin_amdgcn_mfma_f32_16x16x32_f16(ac[i], b[j], acc[i][j], 0, 0, 0); \
        SCHED();                                                                             \
        if (j == 1) { WAIT_L(2); lds_st(freeAddr, (unsigned)(s) + 1u); }                     \
        b[j] = LDSRD(vB + j * 1024);                                                         \
        if (j < 4) an[j] = LDSRD(vA + j * 1024);                                             \
        if (j == 4) fw = *(const volatile LAS u4*)fullAddr;                                  \
        if (j == 7) {                                                                        \
          unsigned m = fw.x < fw.y ? fw.x : fw.y;                                            \
          m = m < fw.z ? m : fw.z;                                                           \
          ready = rfl(m);                                                                    \
        }                                                                                    \
      }                                                                                      \
      SCHED();                                                                               \
    }
    for (int s = 0; s < NST - 2; s += 2) {
      STAGE(a0, a1, s)
      STAGE(a1, a0, s + 1)
    }
    STAGE(a0, a1, NST - 2)
#pragma unroll
    for (int j = 0; j < 8; ++j)
#pragma unroll
      for (int i = 0; i < 4; ++i)
        acc[i][j] = __builtin_amdgcn_mfma_f32_16x16x32_f16(a1[i], b[j], acc[i][j], 0, 0, 0);
#undef STAGE
    if (wave == 0) STAMP(3);

    const unsigned bad = *(const volatile LAS unsigned*)(lds0 + CTRL + 32) |
                         (unsigned)(__hip_atomic_load(tmo, __ATOMIC_RELAXED, __HIP_MEMORY_SCOPE_AGENT) == call);
    float* orow = p.out + (size_t)(rbg * BM + wm * 64 + (lane >> 4) * 4) * N_SH + cb * BN + wn * 128 + (lane & 15) * 4;
    const float nanv = __builtin_nanf("");
#pragma unroll
    for (int i = 0; i < 4; ++i)
#pragma unroll
      for (int r = 0; r < 4; ++r)
#pragma unroll
        for (int g = 0; g < 2; ++g) {
          f4 o = {acc[i][g * 4 + 0][r], acc[i][g * 4 + 1][r], acc[i][g * 4 + 2][r], acc[i][g * 4 + 3][r]};
          if (bad) o = (f4){nanv, nanv, nanv, nanv};
          *(f4*)(orow + (size_t)(i * 16 + r) * N_SH + g * 64) = o;
        }
  } else if (wave < 7) {
    const unsigned call = rfl(__hip_atomic_load(callp, __ATOMIC_RELAXED, __HIP_MEMORY_SCOPE_AGENT)) + 1u;
    if (lane == 0) __hip_atomic_store(callp, call, __ATOMIC_RELAXED, __HIP_MEMORY_SCOPE_AGENT);
    __builtin_amdgcn_s_setprio(2);
    const int ld = wave - 4;
    const char* gsrc;
    const unsigned* flags;
    unsigned gstep, ldst;
    if (ld == 0) {
      gsrc = myws + WS_X16 + (size_t)(call & 1u) * X16_PAR + (size_t)rbg * NST * A_ST;
      flags = (const unsigned*)(myws + WS_FLAGA) + rbg * NST;
      gstep = A_ST;
      ldst = lds0;
    } else {
      gsrc = myws + WS_W16 + (size_t)cb * NST * B_ST + (ld - 1) * 8192;
      flags = (const unsigned*)(myws + WS_FLAGW) + cb * NST;
      gstep = B_ST;
      ldst = lds0 + A_ST + (ld - 1) * 8192;
    }
    gsrc = rfl_ptr(gsrc);
    const unsigned voff = lane * 16;
    const unsigned fullAddr = lds0 + CTRL + ld * 4;
    const unsigned freeAddr = lds0 + CTRL + 16;
    unsigned arrived = 0, freed = 0, pub = 0, slotoff = 0;
    for (unsigned s = 0; s < (unsigned)NST; ++s) {
      if (s >= arrived) {
        WAIT_V(16); if (pub + 2 < s) { pub = s - 2; lds_st(fullAddr, pub); }
        WAIT_V(8);  if (pub + 1 < s) { pub = s - 1; lds_st(fullAddr, pub); }
        WAIT_V(0);  if (pub < s) { pub = s; lds_st(fullAddr, pub); }
        const unsigned fi = s + lane < (unsigned)NST ? s + lane : (unsigned)NST - 1;
        const unsigned long long t0 = __builtin_amdgcn_s_memrealtime();
        for (unsigned spin = 1;; ++spin) {
          const unsigned f = __hip_atomic_load(flags + fi, __ATOMIC_RELAXED, __HIP_MEMORY_SCOPE_SYSTEM);
          const unsigned long long ok = __ballot(f >= call);
          const unsigned n = ok == ~0ull ? 64u : (unsigned)__builtin_ctzll(~ok);
          if (n) { arrived = s + n; break; }
          if ((spin & 63u) == 0u &&
              (rfl(__hip_atomic_load(tmo, __ATOMIC_RELAXED, __HIP_MEMORY_SCOPE_AGENT)) == call ||
               __builtin_amdgcn_s_memrealtime() - t0 > DEADLINE_TICKS)) {
            if (lane == 0) __hip_atomic_store(tmo, call, __ATOMIC_RELAXED, __HIP_MEMORY_SCOPE_AGENT);
            lds_st(lds0 + CTRL + 32, 1u);
            arrived = NST;
            break;
          }
          __builtin_amdgcn_s_sleep(2);
        }
        WAIT_V(0);
      }
      while (freed + NSLOT <= s) {
        u4 f = lds_ld4_wait(freeAddr);
        unsigned m = f.x < f.y ? f.x : f.y, m2 = f.z < f.w ? f.z : f.w;
        freed = rfl(m < m2 ? m : m2);
        if (freed + NSLOT <= s) __builtin_amdgcn_s_sleep(1);
      }
      glds8k(gsrc, voff, ldst + slotoff);
      gsrc += gstep;
      slotoff = slotoff + ST == (unsigned)CTRL ? 0u : slotoff + ST;
      WAIT_V(24);
      if (pub + 2 < s) { pub = s - 2; lds_st(fullAddr, pub); }
    }
    WAIT_V(16); lds_st(fullAddr, NST - 2);
    WAIT_V(8);  lds_st(fullAddr, NST - 1);
    WAIT_V(0);  lds_st(fullAddr, NST);
    if (ld == 0) STAMP(5);
  } else {
    const bool active = bid < NR * NP;
    const int drel = (bid >> 3) & 7;
    const int pj = (bid & 7) + 8 * (bid >> 6);
    const int rb = pj & 3;
    const int ks0 = pj >> 2;
    constexpr int KSTEP = NP / 4;
    constexpr int NT = NST / KSTEP;
    const unsigned prow = lane >> 2, pslot = lane & 3;
    const unsigned pko = pslot ^ ((prow >> 3) << 1);
    unsigned vo[8];
#pragma unroll
    for (int f = 0; f < 8; ++f) vo[f] = (f * 16 + prow) * (KD * 4) + pko * 32;
    const char* xs = rfl_ptr((const char*)p.x + (size_t)rb * BM * KD * 4 + (size_t)ks0 * BK * 4);
    f4 L0[16], L1[16];
#define PLOAD(L, t)                                                                          \
    {                                                                                        \
      const char* sb = xs + (size_t)(t) * (KSTEP * BK * 4);                                  \
      _Pragma("unroll") for (int f = 0; f < 8; ++f) {                                        \
        asm volatile("global_load_dwordx4 %0, %1, %2" : "=v"(L[2 * f]) : "v"(vo[f]), "s"(sb) : "memory"); \
        asm volatile("global_load_dwordx4 %0, %1, %2 offset:16" : "=v"(L[2 * f + 1]) : "v"(vo[f]), "s"(sb) : "memory"); \
      }                                                                                      \
    }
#define PWAIT(L, n)                                                                          \
    asm volatile("s_waitcnt vmcnt(" #n ")"                                                   \
                 : "+v"(L[0]), "+v"(L[1]), "+v"(L[2]), "+v"(L[3]), "+v"(L[4]), "+v"(L[5]), "+v"(L[6]), "+v"(L[7]), \
                   "+v"(L[8]), "+v"(L[9]), "+v"(L[10]), "+v"(L[11]), "+v"(L[12]), "+v"(L[13]), "+v"(L[14]), "+v"(L[15]) \
                 :: "memory")
    if (active) {
      unsigned callv;
      asm volatile("global_load_dword %0, %1, off sc1" : "=v"(callv) : "v"(callp) : "memory");
      PLOAD(L0, 0) PLOAD(L1, 1)
      asm volatile("s_waitcnt vmcnt(32)" : "+v"(callv) :: "memory");
      const unsigned call = rfl(callv) + 1u;
      const unsigned dest = (unsigned)(rank + drel) & 7u;
      char* dws = p.ws[dest < (unsigned)NR ? dest : (unsigned)rank];
      const char* xd = rfl_ptr(dws + WS_X16 + (size_t)(call & 1u) * X16_PAR + ((size_t)(rank * 4 + rb) * NST + ks0) * A_ST);
      const char* fd = rfl_ptr(dws + WS_FLAGA + ((size_t)(rank * 4 + rb) * NST + ks0) * 4);
      const unsigned vs0 = lane * 16, vs1 = lane * 16 + 4096;
#define PSTORE(L, t)                                                                         \
      {                                                                                      \
        const char* db = xd + (size_t)(t) * (KSTEP * A_ST);                                  \
        _Pragma("unroll") for (int f = 0; f < 8; ++f) {                                      \
          u4 o;                                                                              \
          o.x = cvt2(L[2 * f][0], L[2 * f][1]);                                              \
          o.y = cvt2(L[2 * f][2], L[2 * f][3]);                                              \
          o.z = cvt2(L[2 * f + 1][0], L[2 * f + 1][1]);                                      \
          o.w = cvt2(L[2 * f + 1][2], L[2 * f + 1][3]);                                      \
          asm volatile("global_store_dwordx4 %0, %1, %2 offset:%3 sc0 sc1\n\ts_nop 1"        \
                       ::"v"(f < 4 ? vs0 : vs1), "v"(o), "s"(db), "i"((f & 3) * 1024) : "memory"); \
        }                                                                                    \
      }
#define PFLAG(t) flag_store(fd + (size_t)(t) * (KSTEP * 4), 0u, call)
      PWAIT(L0, 16);
      PSTORE(L0, 0) PLOAD(L0, 2)
      PWAIT(L1, 24);
      PSTORE(L1, 1) PLOAD(L1, 3)
      WAIT_V(40);
      PFLAG(0);
      for (int t = 2; t < NT - 2; t += 2) {
        PWAIT(L0, 25);
        PSTORE(L0, t) PLOAD(L0, t + 2)
        WAIT_V(41);
        PFLAG(t - 1);
        PWAIT(L1, 25);
        PSTORE(L1, t + 1) PLOAD(L1, t + 3)
        WAIT_V(41);
        PFLAG(t);
      }
      PWAIT(L0, 25);
      PSTORE(L0, NT - 2)
      WAIT_V(25);
      PFLAG(NT - 3);
      PWAIT(L1, 9);
      PSTORE(L1, NT - 1)
      WAIT_V(9);
      PFLAG(NT - 2);
      WAIT_V(1);
      PFLAG(NT - 1);
      if (lane == 0) __hip_atomic_store(callp, call, __ATOMIC_RELAXED, __HIP_MEMORY_SCOPE_AGENT);
      STAMP(6);
    }
  }
}

}

extern "C" void kernel_launch(int rank, int nranks, ncclComm_t comm,
                              hipStream_t stream,
                              void* const* d_in, const int* in_sizes, int n_in,
                              void* d_out, int out_size,
                              const int* in_dtypes, int out_dtype,
                              void* const* d_out_all,
                              void* const* d_ws_all, size_t ws_bytes) {
  (void)comm; (void)in_dtypes; (void)out_dtype; (void)d_out_all;
  if (nranks != NR || (unsigned)rank >= (unsigned)NR || n_in != 2 || ws_bytes < WS_END ||
      in_sizes[0] != M_SH * KD || in_sizes[1] != KD * N_TOT || out_size != NR * M_SH * N_SH) {
    fprintf(stderr, "kernel_launch: unexpected problem shape\n");
    return;
  }
  static bool attr_set[NR] = {false, false, false, false, false, false, false, false};
  if (!attr_set[rank]) {
    HIP_CHECK(hipFuncSetAttribute((const void*)fused_kernel, hipFuncAttributeMaxDynamicSharedMemorySize, LDS_BYTES));
    attr_set[rank] = true;
  }
  Params p;
  p.x = (const float*)d_in[0];
  p.w = (const float*)d_in[1];
  p.out = (float*)d_out;
  for (int i = 0; i < NR; ++i) p.ws[i] = (char*)d_ws_all[i];
  p.rank = rank;
  p.pad = 0;
  hipLaunchKernelGGL(fused_kernel, dim3(NBLK), dim3(512), LDS_BYTES, stream, p);
  HIP_CHECK(hipGetLastError());
}
```

```cpp
#include <hip/hip_runtime.h>
#include <rccl/rccl.h>
#include <cstdio>

#define HIP_CHECK(expr) do { hipError_t _e = (expr); if (_e != hipSuccess) { \
    fprintf(stderr, "HIP error %s at %s:%d\n", hipGetErrorString(_e), __FILE__, __LINE__); \
    return; } } while (0)

#define LAS __attribute__((address_space(3)))
typedef _Float16 h8 __attribute__((ext_vector_type(8)));
typedef _Float16 h2 __attribute__((ext_vector_type(2)));
typedef float f4 __attribute__((ext_vector_type(4)));
typedef float f2 __attribute__((ext_vector_type(2)));
typedef unsigned u4 __attribute__((ext_vector_type(4)));

namespace {

constexpr int NR = 8;
constexpr int M_SH = 512, KD = 4096, N_SH = 2048, N_TOT = 16384;
constexpr int BM = 128, BN = 256, BK = 32;
constexpr int NST = KD / BK;
constexpr int A_IMG = BM * BK * 2;
constexpr int B_IMG = BN * BK * 2;
constexpr int ST = A_IMG + 1024;
constexpr int NSLOT = 16;
constexpr int CTRL = NSLOT * ST;
constexpr int LDS_BYTES = CTRL + 256;
constexpr int NBLK = 256;
#define PPX 3
constexpr int NPWG = 8 * PPX;
constexpr int NGWG = 232;
constexpr int NITEM = 1280;
constexpr int NUNIT = 4 * NST;
constexpr int PRING = 16;
#define PWIN 3
#define CONV_TICKS 3
constexpr int LOOK = 3;

constexpr size_t WS_CALL = 0;
constexpr size_t WS_TMO = 32768;
constexpr size_t WS_TICKET = 32768 + 256;
constexpr size_t WS_FLAGA = 65536;
constexpr size_t WS_FLAGW = WS_FLAGA + 8 * 4 * 512;
constexpr size_t WS_DBG = 262144;
constexpr size_t WS_X16 = 1u << 20;
constexpr size_t X16_PAR = (size_t)NR * 4 * NST * A_IMG;
constexpr size_t WS_W16 = WS_X16 + 2 * X16_PAR;
constexpr size_t WS_END = WS_W16 + (size_t)8 * NST * B_IMG;

constexpr unsigned long long DEADLINE_TICKS = 40000000ull;

struct Params {
  const float* x;
  const float* w;
  float* out;
  char* ws[NR];
  int rank;
  int pad;
};

#define SCHED() __builtin_amdgcn_sched_barrier(0)
#define CFENCE() asm volatile("" ::: "memory")
#define WAIT_V(n) asm volatile("s_waitcnt vmcnt(%0)" ::"n"(n) : "memory")
#define WAIT_L(n) asm volatile("s_waitcnt lgkmcnt(%0)" ::"n"(n) : "memory")

__device__ __forceinline__ unsigned rfl(unsigned v) { return __builtin_amdgcn_readfirstlane(v); }
__device__ __forceinline__ unsigned umin(unsigned a, unsigned b) { return a < b ? a : b; }
__device__ __forceinline__ unsigned umax(unsigned a, unsigned b) { return a > b ? a : b; }

__device__ __forceinline__ unsigned cvt2(float a, float b) {
  f2 v = {a, b};
  h2 h = __builtin_convertvector(v, h2);
  return __builtin_bit_cast(unsigned, h);
}

__device__ __forceinline__ void glds9k(const char* ga, const char* gb, unsigned voff, unsigned lds_dst) {
  unsigned keep;
  const char* g2 = ga + 4096;
  unsigned d2 = lds_dst + 4096, d3 = lds_dst + 8192;
  asm volatile(
      "s_mov_b32 %0, m0\n\t"
      "s_mov_b32 m0, %5\n\t"
      "s_nop 2\n\t"
      "global_load_lds_dwordx4 %1, %2 sc0 sc1\n\t"
      "global_load_lds_dwordx4 %1, %2 offset:1024 sc0 sc1\n\t"
      "global_load_lds_dwordx4 %1, %2 offset:2048 sc0 sc1\n\t"
      "global_load_lds_dwordx4 %1, %2 offset:3072 sc0 sc1\n\t"
      "s_mov_b32 m0, %6\n\t"
      "s_nop 0\n\t"
      "global_load_lds_dwordx4 %1, %3 sc0 sc1\n\t"
      "global_load_lds_dwordx4 %1, %3 offset:1024 sc0 sc1\n\t"
      "global_load_lds_dwordx4 %1, %3 offset:2048 sc0 sc1\n\t"
      "global_load_lds_dwordx4 %1, %3 offset:3072 sc0 sc1\n\t"
      "s_mov_b32 m0, %7\n\t"
      "s_nop 0\n\t"
      "global_load_lds_dwordx4 %1, %4 sc0 sc1\n\t"
      "s_mov_b32 m0, %0"
      : "=&s"(keep)
      : "v"(voff), "s"(ga), "s"(g2), "s"(gb), "s"(lds_dst), "s"(d2), "s"(d3)
      : "memory");
}

__device__ __forceinline__ void lds_st(unsigned addr, unsigned v) {
  asm volatile("ds_write_b32 %0, %1" ::"v"(addr), "v"(v) : "memory");
}
__device__ __forceinline__ u4 lds_ld4_wait(unsigned addr) {
  u4 r;
  asm volatile("ds_read_b128 %0, %1\n\ts_waitcnt lgkmcnt(0)" : "=&v"(r) : "v"(addr) : "memory");
  return r;
}
__device__ __forceinline__ unsigned lds_ld_wait(unsigned addr) {
  unsigned r;
  asm volatile("ds_read_b32 %0, %1\n\ts_waitcnt lgkmcnt(0)" : "=&v"(r) : "v"(addr) : "memory");
  return r;
}
__device__ __forceinline__ void flag_store(const char* sbase, unsigned voff, unsigned val) {
  unsigned long long keep;
  asm volatile(
      "s_mov_b64 %0, exec\n\t"
      "s_mov_b64 exec, 1\n\t"
      "s_nop 2\n\t"
      "global_store_dword %1, %2, %3 sc0 sc1\n\t"
      "s_mov_b64 exec, %0"
      : "=&s"(keep)
      : "v"(voff), "v"(val), "s"(sbase)
      : "memory");
}
#define STORE16_WT(voff, data, sbase, imm)                                                   \
  asm volatile("global_store_dwordx4 %0, %1, %2 offset:%3 sc0 sc1\n\ts_nop 1"                \
               ::"v"(voff), "v"(data), "s"(sbase), "i"(imm) : "memory")

#define STAMP(i) do {} while (0)

template <int NF>
__device__ __forceinline__ unsigned poll_flags(const unsigned* f0, const unsigned* f1, unsigned call, char* myws,
                                               unsigned badAddr, int lane) {
  unsigned* const tmo = (unsigned*)(myws + WS_TMO);
  const unsigned long long t0 = __builtin_amdgcn_s_memrealtime();
  for (unsigned spin = 1;; ++spin) {
    unsigned f = __hip_atomic_load(f0, __ATOMIC_RELAXED, __HIP_MEMORY_SCOPE_SYSTEM);
    if (NF > 1) f = umin(f, __hip_atomic_load(f1, __ATOMIC_RELAXED, __HIP_MEMORY_SCOPE_SYSTEM));
    const unsigned long long ok = __ballot(f >= call);
    const unsigned n = ok == ~0ull ? 64u : (unsigned)__builtin_ctzll(~ok);
    if (n) return n;
    if ((spin & 63u) == 0u &&
        (rfl(__hip_atomic_load(tmo, __ATOMIC_RELAXED, __HIP_MEMORY_SCOPE_AGENT)) == call ||
         __builtin_amdgcn_s_memrealtime() - t0 > DEADLINE_TICKS)) {
      if (lane == 0) __hip_atomic_store(tmo, call, __ATOMIC_RELAXED, __HIP_MEMORY_SCOPE_AGENT);
      lds_st(badAddr, 1u);
      return 2 * NST;
    }
    __builtin_amdgcn_s_sleep(2);
  }
}

__global__ __launch_bounds__(512, 2) void fused_kernel(Params p) {
  extern __shared__ __attribute__((aligned(1024))) char smem[];
  const unsigned lds0 = (unsigned)(unsigned long long)(LAS char*)smem;
  const int tid = threadIdx.x;
  const int lane = tid & 63;
  const int wave = (int)rfl((unsigned)(tid >> 6));
  const int bid = blockIdx.x;
  const int xcd = bid & 7, idx = bid >> 3;
  const int rank = p.rank;
  char* const myws = p.ws[rank];

  if (wave == 0) STAMP(0);

  unsigned* const callp = (unsigned*)(myws + WS_CALL + (size_t)bid * 128) + wave;

  const unsigned prow = lane >> 2;
  const unsigned pko = (lane & 3) ^ ((prow >> 3) << 1);

  if (tid < 64) ((LAS unsigned*)(smem + CTRL))[tid] = 0u;
  WAIT_L(0);
  __builtin_amdgcn_s_barrier();
  CFENCE();

  if (idx < PPX) {
    const unsigned convAddr = lds0 + CTRL;
    const unsigned sentAddr = lds0 + CTRL + 32;
    const unsigned uidAddr = lds0 + CTRL + 64;
    const unsigned voff = lane * 16;
    if (wave == 0) {
      unsigned vo[8];
#pragma unroll
      for (int f = 0; f < 8; ++f) vo[f] = (f * 16 + prow) * (KD * 4) + pko * 32;
      f4 L0[16], L1[16];
#define PLOAD(L, o)                                                                          \
      {                                                                                      \
        const char* sb = (const char*)p.x + (size_t)((o) & 3u) * (BM * KD * 4) + (size_t)((o) >> 2) * (BK * 4); \
        _Pragma("unroll") for (int f = 0; f < 8; ++f) {                                      \
          L[2 * f] = *(const f4*)(sb + vo[f]);                                               \
          L[2 * f + 1] = *(const f4*)(sb + vo[f] + 16);                                      \
        }                                                                                    \
      }
#define PCONV(L, u, o)                                                                       \
      {                                                                                      \
        const unsigned la = lds0 + ((u) % PRING) * A_IMG + voff;                             \
        _Pragma("unroll") for (int f = 0; f < 8; ++f) {                                      \
          u4 q;                                                                              \
          q.x = cvt2(L[2 * f][0], L[2 * f][1]);                                              \
          q.y = cvt2(L[2 * f][2], L[2 * f][3]);                                              \
          q.z = cvt2(L[2 * f + 1][0], L[2 * f + 1][1]);                                      \
          q.w = cvt2(L[2 * f + 1][2], L[2 * f + 1][3]);                                      \
          *(LAS u4*)(size_t)(la + f * 1024) = q;                                             \
        }                                                                                    \
        lds_st(uidAddr + ((u) % PRING) * 4, (o));                                            \
        WAIT_L(0);                                                                           \
        lds_st(convAddr, (u) + 1u);                                                          \
      }
      const unsigned callv = __hip_atomic_load(callp, __ATOMIC_RELAXED, __HIP_MEMORY_SCOPE_AGENT);
      PLOAD(L0, (unsigned)bid)
      PLOAD(L1, (unsigned)(bid + NPWG))
      SCHED();
      PCONV(L0, 0u, (unsigned)bid)
      PCONV(L1, 1u, (unsigned)(bid + NPWG))
      const unsigned call = rfl(callv) + 1u;
      unsigned* const ticket = (unsigned*)(myws + WS_TICKET);
      const unsigned tbase = (call - 1u) * 1024u;
      if (lane == 0) {
        __hip_atomic_store(callp, call, __ATOMIC_RELAXED, __HIP_MEMORY_SCOPE_AGENT);
        __hip_atomic_fetch_max(ticket, tbase, __ATOMIC_RELAXED, __HIP_MEMORY_SCOPE_AGENT);
      }
      WAIT_V(0);
      for (unsigned u = 2;; ++u) {
        for (;;) {
          const u4 c0 = lds_ld4_wait(sentAddr), c1 = lds_ld4_wait(sentAddr + 16);
          const unsigned mn = rfl(umin(umin(umin(c0.y, c0.z), umin(c0.w, c1.x)), umin(umin(c1.y, c1.z), c1.w)));
          const unsigned mx = rfl(umax(umax(umax(c0.y, c0.z), umax(c0.w, c1.x)), umax(umax(c1.y, c1.z), c1.w)));
          if (mx + 1u >= u && mn + PRING > u) break;
          __builtin_amdgcn_s_sleep(4);
        }
        unsigned tk = 0;
        if (lane == 0) tk = __hip_atomic_fetch_add(ticket, 1u, __ATOMIC_RELAXED, __HIP_MEMORY_SCOPE_AGENT);
        const unsigned o = rfl(tk) - tbase + 2u * NPWG;
        if (o >= (unsigned)NUNIT) {
          lds_st(uidAddr + (u % PRING) * 4, 0xffffffffu);
          WAIT_L(0);
          lds_st(convAddr, u + 1u);
          break;
        }
        PLOAD(L0, o)
        PCONV(L0, u, o)
      }
      STAMP(16);
    } else {
      const unsigned call = rfl(__hip_atomic_load(callp, __ATOMIC_RELAXED, __HIP_MEMORY_SCOPE_AGENT)) + 1u;
      if (lane == 0) __hip_atomic_store(callp, call, __ATOMIC_RELAXED, __HIP_MEMORY_SCOPE_AGENT);
      const unsigned dest = (unsigned)(rank + wave) & 7u;
      char* dws = p.ws[dest < (unsigned)NR ? dest : (unsigned)rank];
      const char* xd = dws + WS_X16 + (size_t)(call & 1u) * X16_PAR + (size_t)(rank * 4) * NST * A_IMG;
      const char* fd = dws + WS_FLAGA + (size_t)(rank * 4) * NST * 4;
      const unsigned voff2 = voff + 4096;
      unsigned conv = 0, prev = 0;
      for (unsigned u = 0;; ++u) {
        while (conv <= u) {
          conv = rfl(lds_ld_wait(convAddr));
          if (conv <= u) __builtin_amdgcn_s_sleep(1);
        }
        const unsigned o = rfl(lds_ld_wait(uidAddr + (u % PRING) * 4));
        if (o >= (unsigned)NUNIT) break;
        const unsigned la = lds0 + (u % PRING) * A_IMG + voff;
        u4 d[8];
#pragma unroll
        for (int f = 0; f < 8; ++f) d[f] = *(const LAS u4*)(size_t)(la + f * 1024);
        asm volatile("s_waitcnt lgkmcnt(0)" : "+v"(d[0]), "+v"(d[1]), "+v"(d[2]), "+v"(d[3]), "+v"(d[4]), "+v"(d[5]),
                     "+v"(d[6]), "+v"(d[7]) :: "memory");
        lds_st(sentAddr + wave * 4, u + 1u);
        const unsigned im = (o & 3u) * NST + (o >> 2);
        const char* db = xd + (size_t)im * A_IMG;
#pragma unroll
        for (int f = 0; f < 8; ++f) {
          WAIT_V(PWIN - 1);
          if (f == PWIN - 1 && u > 0)
            flag_store(fd + (size_t)prev * 4, 0u, call);
          STORE16_WT(f < 4 ? voff : voff2, d[f], db, (f & 3) * 1024);
        }
        prev = im;
      }
      WAIT_V(0);
      flag_store(fd + (size_t)prev * 4, 0u, call);
      STAMP(16 + wave);
    }
    return;
  }

  const int gl = (xcd >> 1) * 29 + (idx - PPX);
  const int tr = gl >> 2;
  const int cb = (xcd & 1) * 4 + (gl & 3);
  const int gidx = (idx - PPX) * 8 + xcd;
  const int blk = tr < 28 ? (((rank + 1 + (tr >> 2)) & 7) * 4 + (tr & 3)) : rank * 4 + 3;
  const int lfrag = tr < 28 ? tr : 0;
  const unsigned fullAddr = lds0 + CTRL;
  const unsigned freeAddr = lds0 + CTRL + 16;
  const unsigned badAddr = lds0 + CTRL + 32;

  if (wave == 5 || wave == 6) return;
  const unsigned long long tstart = __builtin_amdgcn_s_memrealtime();
  const unsigned call = rfl(__hip_atomic_load(callp, __ATOMIC_RELAXED, __HIP_MEMORY_SCOPE_AGENT)) + 1u;
  if (lane == 0) __hip_atomic_store(callp, call, __ATOMIC_RELAXED, __HIP_MEMORY_SCOPE_AGENT);

  if (wave < 4) {
    const unsigned arow = lane & 15, ako = lane >> 4;
    const unsigned offA = lds0 + arow * 64 + ((ako ^ ((arow >> 3) << 1)) << 4);
    const unsigned myFree = freeAddr + wave * 4;
    const __amdgpu_buffer_rsrc_t wrs =
        __builtin_amdgcn_make_buffer_rsrc(myws + WS_W16 + (size_t)cb * NST * B_IMG, 0, NST * B_IMG, 0x00020000);
    const unsigned voffB = wave * 4096 + lane * 16;

    f4 acc[9][4];
#pragma unroll
    for (int i = 0; i < 9; ++i)
#pragma unroll
      for (int j = 0; j < 4; ++j) acc[i][j] = (f4){0.f, 0.f, 0.f, 0.f};
    h8 a[6];
    u4 b[4][4];

#define LDSRD(addr) (*(const LAS h8*)(size_t)(addr))
#define LDB(ks, j) __builtin_amdgcn_raw_buffer_load_b128(wrs, voffB + (j) * 1024, (ks) * B_IMG, 17)
#define MFMA(c, av, bv) asm volatile("v_mfma_f32_16x16x32_f16 %0, %1, %2, %0" : "+v"(c) : "v"(av), "v"(bv))
    unsigned ready = 0;
    auto wait_ready = [&](unsigned need) {
      while (ready < need) {
        ready = rfl(lds_ld_wait(fullAddr));
        if (ready < need) __builtin_amdgcn_s_sleep(1);
      }
    };
    wait_ready(1);
    if (wave == 0) STAMP(2);
#pragma unroll
    for (int t = 0; t < 3; ++t)
#pragma unroll
      for (int j = 0; j < 4; ++j) b[t][j] = LDB(t, j);
#pragma unroll
    for (int i = 0; i < 6; ++i) a[i] = LDSRD(offA + i * 1024);
    asm volatile("s_nop 1" : "+v"(a[0]), "+v"(a[1]), "+v"(a[2]), "+v"(a[3]), "+v"(a[4]), "+v"(a[5]));

    unsigned slotoff = 0;
    unsigned vA = offA;
    for (unsigned s4 = 0; s4 < (unsigned)NST; s4 += 4) {
#pragma unroll
      for (int t = 0; t < 4; ++t) {
        const unsigned s = s4 + t;
        unsigned fw, vN;
#pragma unroll
        for (int i = 0; i < 9; ++i) {
          const int buf = (t * 9 + i) % 6;
          SCHED();
#pragma unroll
          for (int j = 0; j < 4; ++j) MFMA(acc[i][j], a[buf], b[t][j]);
          SCHED();
          if (i < 3) {
            a[buf] = LDSRD(vA + (i + 6) * 1024);
          } else {
            if (i == 3) {
              if (ready < s + 2u) wait_ready(s + 2u);
              CFENCE();
            }
            if (i == 5) { WAIT_L(2); lds_st(myFree, s + 1u); }
            a[buf] = LDSRD(vN + (i - 3) * 1024);
          }
          if (i == 0) fw = *(const volatile LAS unsigned*)(size_t)fullAddr;
          if (i == 1) {
            slotoff = slotoff + ST == (unsigned)CTRL ? 0u : slotoff + ST;
            vN = offA + slotoff;
          }
          if (i == 2) ready = rfl(fw);
          if (i >= 4 && i < 8) b[(t + 3) & 3][i - 4] = LDB(s + 3u, i - 4);
        }
        vA = vN;
      }
    }
    asm volatile("s_nop 7\n\ts_nop 7" ::: "memory");
    SCHED();
    if (wave == 0) STAMP(3);

    unsigned* const tmo = (unsigned*)(myws + WS_TMO);
    const unsigned bad = *(const volatile LAS unsigned*)(size_t)badAddr |
                         (unsigned)(__hip_atomic_load(tmo, __ATOMIC_RELAXED, __HIP_MEMORY_SCOPE_AGENT) == call);
    float* ocol = p.out + (size_t)((lane >> 4) * 4) * N_SH + cb * BN + wave * 64 + (lane & 15) * 4;
    const float nanv = __builtin_nanf("");
#pragma unroll
    for (int i = 0; i < 9; ++i) {
      if (tr == 28 && (i < 4 || i == 8)) continue;
      float* orow = ocol + (size_t)(i < 8 ? blk * BM + i * 16 : rank * M_SH + lfrag * 16) * N_SH;
#pragma unroll
      for (int r = 0; r < 4; ++r) {
        f4 o = {acc[i][0][r], acc[i][1][r], acc[i][2][r], acc[i][3][r]};
        if (bad) o = (f4){nanv, nanv, nanv, nanv};
        asm volatile("global_store_dwordx4 %0, %1, off sc0 sc1\n\ts_nop 1" ::"v"(orow + (size_t)r * N_SH), "v"(o) : "memory");
      }
    }
  } else if (wave == 4) {
    __builtin_amdgcn_s_setprio(2);
    const char* x16 = myws + WS_X16 + (size_t)(call & 1u) * X16_PAR;
    const int lblk = rank * 4 + (lfrag >> 3);
    const char* ga = x16 + (size_t)blk * NST * A_IMG;
    const char* gb = x16 + (size_t)lblk * NST * A_IMG + (lfrag & 7) * 1024;
    const unsigned* fa = (const unsigned*)(myws + WS_FLAGA) + blk * NST;
    const unsigned* fl = (const unsigned*)(myws + WS_FLAGA) + lblk * NST;
    const unsigned* fw = (const unsigned*)(myws + WS_FLAGW) + cb * NST;
    const unsigned voff = lane * 16;
    unsigned arrA = 0;
    unsigned arrL = 0;
    unsigned freed = 0, pub = 0, slotoff = 0;
    for (unsigned s = 0; s < (unsigned)NST; ++s) {
      const unsigned needL = umin(s + 1u + LOOK, NST);
      if (s >= arrA || arrL < needL) {
        WAIT_V(18); if (pub + 2 < s) { pub = s - 2; lds_st(fullAddr, pub); }
        WAIT_V(9);  if (pub + 1 < s) { pub = s - 1; lds_st(fullAddr, pub); }
        WAIT_V(0);  if (pub < s) { pub = s; lds_st(fullAddr, pub); }
        while (arrL < needL) {
          const unsigned fi = umin(arrL + lane, NST - 1);
          arrL += poll_flags<2>(fl + fi, fw + fi, call, myws, badAddr, lane);
        }
        if (s >= arrA) arrA = s + poll_flags<1>(fa + umin(s + lane, NST - 1), nullptr, call, myws, badAddr, lane);
        WAIT_V(0);
      }
      while (freed + NSLOT <= s) {
        const u4 f = lds_ld4_wait(freeAddr);
        freed = rfl(umin(umin(f.x, f.y), umin(f.z, f.w)));
        if (freed + NSLOT <= s) __builtin_amdgcn_s_sleep(2);
      }
      glds9k(ga, gb, voff, lds0 + slotoff);
      ga += A_IMG;
      gb += A_IMG;
      slotoff = slotoff + ST == (unsigned)CTRL ? 0u : slotoff + ST;
      WAIT_V(27);
      if (pub + 2 < s) { pub = s - 2; lds_st(fullAddr, pub); }
    }
    WAIT_V(18); lds_st(fullAddr, NST - 2);
    WAIT_V(9);  lds_st(fullAddr, NST - 1);
    WAIT_V(0);  lds_st(fullAddr, 2 * NST);
    STAMP(5);
  } else {
    const unsigned voff = lane * 16;
    for (int item = gidx; item < NITEM; item += NGWG) {
      while (__builtin_amdgcn_s_memrealtime() - tstart < (unsigned long long)(item * CONV_TICKS)) __builtin_amdgcn_s_sleep(8);
      const int pair = item / 20, r = item % 20;
      if (r < 16) {
        const int wcb = r & 7, wks = pair * 2 + (r >> 3);
        const float* wsrc = p.w + (size_t)(wks * BK + (lane >> 4) * 8) * N_TOT + (size_t)rank * N_SH + wcb * BN + (lane & 15) * 4;
        f4 v[4][8];
#pragma unroll
        for (int g = 0; g < 4; ++g)
#pragma unroll
          for (int t = 0; t < 8; ++t) v[g][t] = *(const f4*)(wsrc + (size_t)t * N_TOT + g * 64);
        SCHED();
        const char* dst = myws + WS_W16 + ((size_t)wcb * NST + wks) * B_IMG;
#pragma unroll
        for (int g = 0; g < 4; ++g) {
          const char* dg = dst + g * 4096;
#pragma unroll
          for (int j = 0; j < 4; ++j) {
            u4 o;
            o.x = cvt2(v[g][0][j], v[g][1][j]);
            o.y = cvt2(v[g][2][j], v[g][3][j]);
            o.z = cvt2(v[g][4][j], v[g][5][j]);
            o.w = cvt2(v[g][6][j], v[g][7][j]);
            STORE16_WT(voff, o, dg, j * 1024);
          }
        }
        WAIT_V(0);
        flag_store(myws + WS_FLAGW + (size_t)(wcb * NST + wks) * 4, 0u, call);
      } else {
        const int rb = r - 16, ks = pair * 2;
        const float* xsrc = p.x + (size_t)(rb * BM + prow) * KD + ks * BK + pko * 8;
        f4 v[2][8][2];
#pragma unroll
        for (int f = 0; f < 8; ++f)
#pragma unroll
          for (int h = 0; h < 2; ++h) {
            v[h][f][0] = *(const f4*)(xsrc + (size_t)f * 16 * KD + h * BK);
            v[h][f][1] = *(const f4*)(xsrc + (size_t)f * 16 * KD + h * BK + 4);
          }
        SCHED();
        const char* dst = myws + WS_X16 + (size_t)(call & 1u) * X16_PAR + ((size_t)(rank * 4 + rb) * NST + ks) * A_IMG;
#pragma unroll
        for (int h = 0; h < 2; ++h)
#pragma unroll
          for (int f = 0; f < 8; ++f) {
            u4 o;
            o.x = cvt2(v[h][f][0][0], v[h][f][0][1]);
            o.y = cvt2(v[h][f][0][2], v[h][f][0][3]);
            o.z = cvt2(v[h][f][1][0], v[h][f][1][1]);
            o.w = cvt2(v[h][f][1][2], v[h][f][1][3]);
            const char* dg = dst + h * A_IMG + (f >> 2) * 4096;
            STORE16_WT(voff, o, dg, (f & 3) * 1024);
          }
        WAIT_V(0);
        const char* fl = myws + WS_FLAGA + ((size_t)(rank * 4 + rb) * NST + ks) * 4;
        flag_store(fl, 0u, call);
        flag_store(fl + 4, 0u, call);
      }
    }
    STAMP(1);
  }
}

}

extern "C" void kernel_launch(int rank, int nranks, ncclComm_t comm,
                              hipStream_t stream,
                              void* const* d_in, const int* in_sizes, int n_in,
                              void* d_out, int out_size,
                              const int* in_dtypes, int out_dtype,
                              void* const* d_out_all,
                              void* const* d_ws_all, size_t ws_bytes) {
  (void)comm; (void)in_dtypes; (void)out_dtype; (void)d_out_all;
  if (nranks != NR || (unsigned)rank >= (unsigned)NR || n_in != 2 || ws_bytes < WS_END ||
      in_sizes[0] != M_SH * KD || in_sizes[1] != KD * N_TOT || out_size != NR * M_SH * N_SH) {
    fprintf(stderr, "kernel_launch: unexpected problem shape\n");
    return;
  }
  static bool attr_set[NR] = {false, false, false, false, false, false, false, false};
  if (!attr_set[rank]) {
    HIP_CHECK(hipFuncSetAttribute((const void*)fused_kernel, hipFuncAttributeMaxDynamicSharedMemorySize, LDS_BYTES));
    attr_set[rank] = true;
  }
  Params p;
  p.x = (const float*)d_in[0];
  p.w = (const float*)d_in[1];
  p.out = (float*)d_out;
  for (int i = 0; i < NR; ++i) p.ws[i] = (char*)d_ws_all[i];
  p.rank = rank;
  p.pad = 0;
  hipLaunchKernelGGL(fused_kernel, dim3(NBLK), dim3(512), LDS_BYTES, stream, p);
  HIP_CHECK(hipGetLastError());
}
```

```cpp
#include <hip/hip_runtime.h>
#include <rccl/rccl.h>
#include <cstdio>

#define HIP_CHECK(expr) do { hipError_t _e = (expr); if (_e != hipSuccess) { \
    fprintf(stderr, "HIP error %s at %s:%d\n", hipGetErrorString(_e), __FILE__, __LINE__); \
    return; } } while (0)

#define LAS __attribute__((address_space(3)))
typedef _Float16 h8 __attribute__((ext_vector_type(8)));
typedef _Float16 h2 __attribute__((ext_vector_type(2)));
typedef float f4 __attribute__((ext_vector_type(4)));
typedef float f2 __attribute__((ext_vector_type(2)));
typedef unsigned u4 __attribute__((ext_vector_type(4)));
typedef unsigned u2 __attribute__((ext_vector_type(2)));
typedef unsigned short us2 __attribute__((ext_vector_type(2)));

namespace {

constexpr int NR = 8;
constexpr int M_SH = 512, KD = 4096, N_SH = 2048, N_TOT = 16384;
constexpr int BM = 128, BN = 256, BK = 32;
constexpr int NST = KD / BK;
constexpr int A_IMG = BM * BK * 2;
constexpr int B_IMG = BN * BK * 2;
constexpr int ST = A_IMG + 1024;
constexpr int NSLOT = 16;
constexpr int CTRL = NSLOT * ST;
constexpr int LDS_BYTES = CTRL + 256;
constexpr int NBLK = 256;
#define PPX 3
#define NPACT (8 * PPX)
constexpr int NPWG = NPACT;
constexpr int NGWG = 232;
constexpr int NITEM = 1280;
constexpr int NUNIT = 4 * NST;
constexpr int PRING = 16;
#define PWIN 3
#define PLEAD 0u
#define BPOS 0
#define LEAD0 0
#define LEAD1 0
#define OPENWIN 0
#define POLL_SLEEP 1
#define RAWTAIL 2
#define CPRIO 0
#define FPRIO 1
#define EPI_BITS " sc0 sc1"
#define CONV_TICKS 3
constexpr int LOOK = 3;

constexpr size_t WS_CALL = 0;
constexpr size_t WS_TMO = 32768;
constexpr size_t WS_TICKET = 32768 + 256;
constexpr size_t WS_FLAGA = 65536;
constexpr size_t FLAGA_PAR = (size_t)NR * 4 * NST * 8;
constexpr size_t WS_FLAGW = WS_FLAGA + 2 * FLAGA_PAR;
constexpr size_t WS_FLAGL = WS_FLAGW + 8 * NST * 4;
constexpr size_t WS_DBG = 262144;
constexpr size_t WS_X16 = 1u << 20;
constexpr size_t X16_PAR = (size_t)NR * 4 * NST * A_IMG;
constexpr size_t WS_W16 = WS_X16 + 2 * X16_PAR;
constexpr size_t WS_END = WS_W16 + (size_t)8 * NST * B_IMG;

constexpr unsigned META_RAW = 0x80000000u;
constexpr int EXC_OFF = 7168;
constexpr unsigned MAXEXC = 128;

constexpr unsigned long long DEADLINE_TICKS = 40000000ull;

struct Params {
  const float* x;
  const float* w;
  float* out;
  char* ws[NR];
  int rank;
  int pad;
};

#define SCHED() __builtin_amdgcn_sched_barrier(0)
#define CFENCE() asm volatile("" ::: "memory")
#define WAIT_V(n) asm volatile("s_waitcnt vmcnt(%0)" ::"n"(n) : "memory")
#define WAIT_L(n) asm volatile("s_waitcnt lgkmcnt(%0)" ::"n"(n) : "memory")

__device__ __forceinline__ unsigned rfl(unsigned v) { return __builtin_amdgcn_readfirstlane(v); }
__device__ __forceinline__ unsigned umin(unsigned a, unsigned b) { return a < b ? a : b; }
__device__ __forceinline__ unsigned umax(unsigned a, unsigned b) { return a > b ? a : b; }

__device__ __forceinline__ unsigned cvt2(float a, float b) {
  f2 v = {a, b};
  h2 h = __builtin_convertvector(v, h2);
  return __builtin_bit_cast(unsigned, h);
}

__device__ __forceinline__ unsigned image_of(unsigned o) {
  constexpr unsigned MID = NST - LEAD0 - LEAD1;
  if (o < LEAD0) return o;
  o -= LEAD0;
  if (o < 2u * LEAD1) return (o & 1u) ? NST + (o >> 1) : LEAD0 + (o >> 1);
  o -= 2u * LEAD1;
  if (o < 4u * MID) {
    const unsigned rb = o & 3u;
    return rb * NST + (o >> 2) + (rb == 0u ? LEAD0 + LEAD1 : rb == 1u ? LEAD1 : 0u);
  }
  o -= 4u * MID;
  if (o < 3u * LEAD0) {
    const unsigned rb = 1u + o % 3u;
    return rb * NST + MID + o / 3u + (rb == 1u ? LEAD1 : 0u);
  }
  o -= 3u * LEAD0;
  return (2u + (o & 1u)) * NST + NST - LEAD1 + (o >> 1);
}

__device__ __forceinline__ void lds_st(unsigned addr, unsigned v) {
  asm volatile("ds_write_b32 %0, %1" ::"v"(addr), "v"(v) : "memory");
}
__device__ __forceinline__ u4 lds_ld4_wait(unsigned addr) {
  u4 r;
  asm volatile("ds_read_b128 %0, %1\n\ts_waitcnt lgkmcnt(0)" : "=&v"(r) : "v"(addr) : "memory");
  return r;
}
__device__ __forceinline__ unsigned lds_ld_wait(unsigned addr) {
  unsigned r;
  asm volatile("ds_read_b32 %0, %1\n\ts_waitcnt lgkmcnt(0)" : "=&v"(r) : "v"(addr) : "memory");
  return r;
}
__device__ __forceinline__ void flag_store(const char* sbase, unsigned voff, unsigned val) {
  unsigned long long keep;
  asm volatile(
      "s_mov_b64 %0, exec\n\t"
      "s_mov_b64 exec, 1\n\t"
      "s_nop 2\n\t"
      "global_store_dword %1, %2, %3 sc0 sc1\n\t"
      "s_mov_b64 exec, %0"
      : "=&s"(keep)
      : "v"(voff), "v"(val), "s"(sbase)
      : "memory");
}
__device__ __forceinline__ void flag_store2(const char* sbase, unsigned voff, unsigned lo, unsigned hi) {
  unsigned long long keep;
  const u2 v = {lo, hi};
  asm volatile(
      "s_mov_b64 %0, exec\n\t"
      "s_mov_b64 exec, 1\n\t"
      "s_nop 2\n\t"
      "global_store_dwordx2 %1, %2, %3 sc0 sc1\n\t"
      "s_mov_b64 exec, %0"
      : "=&s"(keep)
      : "v"(voff), "v"(v), "s"(sbase)
      : "memory");
}
__device__ __forceinline__ unsigned perm(unsigned hi, unsigned lo, unsigned sel) { return __builtin_amdgcn_perm(hi, lo, sel); }
__device__ __forceinline__ unsigned xad(unsigned a, unsigned b, unsigned c) {
  unsigned r;
  asm("v_xad_u32 %0, %1, %2, %3" : "=v"(r) : "v"(a), "v"(b), "s"(c));
  return r;
}

__device__ __forceinline__ unsigned encode_image(const unsigned (&D)[32], unsigned slot, int lane, bool force_raw) {
  const unsigned la = slot + lane * 16;
  unsigned meta = META_RAW;
  if (!force_raw) {
    us2 mx = {0, 0};
#pragma unroll
    for (int i = 0; i < 32; ++i) mx = __builtin_elementwise_max(mx, __builtin_bit_cast(us2, D[i] & 0x7fff7fffu));
    const unsigned el = (unsigned)(mx.x > mx.y ? mx.x : mx.y) >> 10;
    unsigned lo = 7, hi = 31;
    while (lo < hi) {
      const unsigned mid = (lo + hi) >> 1;
      if (__builtin_popcountll(__ballot(el > mid)) <= 20) hi = mid; else lo = mid + 1;
    }
    const unsigned base = lo - 7;
    const unsigned bb = base * 0x04040404u;
    unsigned cnt = 0;
    u4 Lp[4];
    unsigned P[12];
#pragma unroll
    for (int g = 0; g < 4; ++g) {
      unsigned T[4], Q = 0;
#pragma unroll
      for (int k = 0; k < 4; ++k) {
        const unsigned d0 = D[8 * g + 2 * k], d1 = D[8 * g + 2 * k + 1];
        Lp[g][k] = perm(d1, d0, 0x06040200u);
        const unsigned H = perm(d1, d0, 0x07050301u);
        const unsigned R = (H | 0x80808080u) - bb;
        T[k] = (R & 0x1f1f1f1fu) | (H & 0x80808080u);
        const unsigned dl = H ^ (T[k] + bb);
        Q |= (dl >> 5) << (2 * k);
      }
      P[3 * g + 0] = T[0] | ((T[3] << 5) & 0x60606060u);
      P[3 * g + 1] = T[1] | ((T[3] << 3) & 0x60606060u);
      P[3 * g + 2] = T[2] | ((T[3] << 1) & 0x20202020u) | ((T[3] >> 1) & 0x40404040u);
      for (;;) {
        const unsigned long long m = __ballot(Q != 0u);
        if (m == 0ull || cnt > MAXEXC) break;
        if (Q != 0u) {
          const unsigned pos = (unsigned)__builtin_ctz(Q) & ~1u;
          const unsigned dv = (Q >> pos) & 3u;
          Q &= ~(3u << pos);
          const unsigned n = ((pos >> 1) & 3u) * 4u + (pos >> 3);
          const unsigned hw = (2u * g + (n >> 3)) * 512u + (unsigned)lane * 8u + (n & 7u);
          const unsigned ix = cnt + __builtin_amdgcn_mbcnt_hi((unsigned)(m >> 32), __builtin_amdgcn_mbcnt_lo((unsigned)m, 0u));
          *(LAS unsigned short*)(size_t)(slot + EXC_OFF + ix * 2u) = (unsigned short)(hw | (dv << 12));
        }
        cnt += (unsigned)__builtin_popcountll(m);
      }
    }
    if (cnt <= MAXEXC) {
      meta = base | (cnt << 8);
#pragma unroll
      for (int g = 0; g < 4; ++g) *(LAS u4*)(size_t)(la + g * 1024) = Lp[g];
#pragma unroll
      for (int t = 0; t < 3; ++t)
        *(LAS u4*)(size_t)(la + (4 + t) * 1024) = (u4){P[4 * t], P[4 * t + 1], P[4 * t + 2], P[4 * t + 3]};
    }
  }
  if (meta == META_RAW) {
#pragma unroll
    for (int f = 0; f < 8; ++f)
      *(LAS u4*)(size_t)(la + f * 1024) = (u4){D[4 * f], D[4 * f + 1], D[4 * f + 2], D[4 * f + 3]};
  }
  return meta;
}

__device__ __forceinline__ void decode_image(const u4 (&R)[9], unsigned e0, unsigned e1, unsigned slot, unsigned meta,
                                             int lane) {
  const unsigned la = slot + lane * 16;
  const unsigned bb = (meta & 31u) * 0x04040404u;
  const unsigned cnt = meta >> 8;
#pragma unroll
  for (int g = 0; g < 4; ++g) {
    unsigned H[4], X[3];
#pragma unroll
    for (int k = 0; k < 3; ++k) {
      const unsigned P = R[4 + ((3 * g + k) >> 2)][(3 * g + k) & 3];
      X[k] = P & 0x60606060u;
      H[k] = xad(P, X[k], bb);
    }
    const unsigned a = (X[0] >> 5) | (X[1] >> 3) | (X[2] >> 1);
    const unsigned sg = a & 0x20202020u;
    H[3] = xad(a, sg, bb) | (sg << 2);
#pragma unroll
    for (int h = 0; h < 2; ++h) {
      u4 o;
      o.x = perm(H[2 * h], R[g][2 * h], 0x05010400u);
      o.y = perm(H[2 * h], R[g][2 * h], 0x07030602u);
      o.z = perm(H[2 * h + 1], R[g][2 * h + 1], 0x05010400u);
      o.w = perm(H[2 * h + 1], R[g][2 * h + 1], 0x07030602u);
      asm volatile("ds_write_b128 %0, %1 offset:%2" ::"v"(la), "v"(o), "i"((2 * g + h) * 1024) : "memory");
    }
  }
  {
    const unsigned e = (unsigned)lane < cnt ? e0 : 0u;
    asm volatile("ds_xor_b32 %0, %1" ::"v"(slot + ((e & 0xffeu) << 1)), "v"((e >> 12) << (13u + ((e & 1u) << 4))) : "memory");
  }
  if (cnt > 64u) {
    const unsigned e = (unsigned)lane + 64u < cnt ? e1 : 0u;
    asm volatile("ds_xor_b32 %0, %1" ::"v"(slot + ((e & 0xffeu) << 1)), "v"((e >> 12) << (13u + ((e & 1u) << 4))) : "memory");
  }
}

#define STORE16_WT(voff, data, sbase, imm)                                                   \
  asm volatile("global_store_dwordx4 %0, %1, %2 offset:%3 sc0 sc1\n\ts_nop 1"                \
               ::"v"(voff), "v"(data), "s"(sbase), "i"(imm) : "memory")

#define STAMP(i) do {} while (0)

__device__ __forceinline__ unsigned poll_flags(const unsigned* f0, const unsigned* f1, unsigned call, char* myws,
                                               unsigned badAddr, int lane) {
  unsigned* const tmo = (unsigned*)(myws + WS_TMO);
  const unsigned long long t0 = __builtin_amdgcn_s_memrealtime();
  for (unsigned spin = 1;; ++spin) {
    const unsigned f = umin(__hip_atomic_load(f0, __ATOMIC_RELAXED, __HIP_MEMORY_SCOPE_SYSTEM),
                            __hip_atomic_load(f1, __ATOMIC_RELAXED, __HIP_MEMORY_SCOPE_SYSTEM));
    const unsigned long long ok = __ballot(f >= call);
    const unsigned n = ok == ~0ull ? 64u : (unsigned)__builtin_ctzll(~ok);
    if (n) return n;
    if ((spin & 63u) == 0u &&
        (rfl(__hip_atomic_load(tmo, __ATOMIC_RELAXED, __HIP_MEMORY_SCOPE_AGENT)) == call ||
         __builtin_amdgcn_s_memrealtime() - t0 > DEADLINE_TICKS)) {
      if (lane == 0) __hip_atomic_store(tmo, call, __ATOMIC_RELAXED, __HIP_MEMORY_SCOPE_AGENT);
      lds_st(badAddr, 1u);
      return 64u;
    }
    __builtin_amdgcn_s_sleep(2);
  }
}

__global__ __launch_bounds__(512, 2) void fused_kernel(Params p) {
  extern __shared__ __attribute__((aligned(1024))) char smem[];
  const unsigned lds0 = (unsigned)(unsigned long long)(LAS char*)smem;
  const int tid = threadIdx.x;
  const int lane = tid & 63;
  const int wave = (int)rfl((unsigned)(tid >> 6));
  const int bid = blockIdx.x;
  const int xcd = bid & 7, idx = bid >> 3;
  const int rank = p.rank;
  char* const myws = p.ws[rank];

  if (wave == 0) STAMP(0);

  unsigned* const callp = (unsigned*)(myws + WS_CALL + (size_t)bid * 128) + wave;

  const unsigned prow = lane >> 2;
  const unsigned pko = (lane & 3) ^ ((prow >> 3) << 1);

  if (tid < 64) ((LAS unsigned*)(smem + CTRL))[tid] = 0u;
  WAIT_L(0);
  __builtin_amdgcn_s_barrier();
  CFENCE();

  if (idx < PPX) {
    if (bid >= NPWG) return;
    const unsigned convAddr = lds0 + CTRL;
    const unsigned sentAddr = lds0 + CTRL + 32;
    const unsigned uidAddr = lds0 + CTRL + 64;
    const unsigned voff = lane * 16;
    if (wave == 0) {
      unsigned vo[8];
#pragma unroll
      for (int f = 0; f < 8; ++f) vo[f] = (f * 16 + prow) * (KD * 4) + pko * 32;
      f4 L0[16], L1[16];
#define PLOAD(L, o)                                                                          \
      {                                                                                      \
        const unsigned im_ = image_of(o);                                                    \
        const char* sb = (const char*)p.x + (size_t)(im_ / NST) * (BM * KD * 4) + (size_t)(im_ % NST) * (BK * 4); \
        _Pragma("unroll") for (int f = 0; f < 8; ++f) {                                      \
          L[2 * f] = *(const f4*)(sb + vo[f]);                                               \
          L[2 * f + 1] = *(const f4*)(sb + vo[f] + 16);                                      \
        }                                                                                    \
      }
#define PCVT(D, L)                                                                           \
      _Pragma("unroll") for (int f = 0; f < 8; ++f) {                                        \
        D[4 * f + 0] = cvt2(L[2 * f][0], L[2 * f][1]);                                       \
        D[4 * f + 1] = cvt2(L[2 * f][2], L[2 * f][3]);                                       \
        D[4 * f + 2] = cvt2(L[2 * f + 1][0], L[2 * f + 1][1]);                               \
        D[4 * f + 3] = cvt2(L[2 * f + 1][2], L[2 * f + 1][3]);                               \
      }
#define PPUB(D, u, o, raw)                                                                   \
      {                                                                                      \
        const unsigned meta = encode_image(D, lds0 + ((u) % PRING) * A_IMG, lane, raw);      \
        lds_st(uidAddr + ((u) % PRING) * 8, (o));                                            \
        lds_st(uidAddr + ((u) % PRING) * 8 + 4, meta);                                       \
        WAIT_L(0);                                                                           \
        lds_st(convAddr, (u) + 1u);                                                          \
      }
#define PTICKET(tk)                                                                          \
      {                                                                                      \
        unsigned long long keep;                                                             \
        asm volatile("s_mov_b64 %1, exec\n\ts_mov_b64 exec, 1\n\ts_nop 2\n\t"                \
                     "global_atomic_add %0, %2, %3, %4 sc0\n\ts_mov_b64 exec, %1"            \
                     : "=&v"(tk), "=&s"(keep) : "v"(0u), "v"(1u), "s"(ticket) : "memory");   \
      }
      constexpr bool kRaw = false;
      const unsigned callv = __hip_atomic_load(callp, __ATOMIC_RELAXED, __HIP_MEMORY_SCOPE_AGENT);
      PLOAD(L1, (unsigned)bid)
      PLOAD(L0, (unsigned)(bid + NPWG))
      SCHED();
      unsigned D[32];
      PCVT(D, L1)
      PPUB(D, 0u, (unsigned)bid, true)
      const unsigned call = rfl(callv) + 1u;
      unsigned* const ticket = (unsigned*)(myws + WS_TICKET);
      const unsigned tbase = (call - 1u) * 1024u;
      if (lane == 0) {
        __hip_atomic_store(callp, call, __ATOMIC_RELAXED, __HIP_MEMORY_SCOPE_AGENT);
        __hip_atomic_fetch_max(ticket, tbase, __ATOMIC_RELAXED, __HIP_MEMORY_SCOPE_AGENT);
      }
      WAIT_V(0);
      __builtin_amdgcn_s_setprio(1);
      unsigned o = bid + NPWG, tk;
      PTICKET(tk)
      for (unsigned u = 1;; ++u) {
        for (;;) {
          const u4 c0 = lds_ld4_wait(sentAddr), c1 = lds_ld4_wait(sentAddr + 16);
          const unsigned mn = rfl(umin(umin(umin(c0.y, c0.z), umin(c0.w, c1.x)), umin(umin(c1.y, c1.z), c1.w)));
          const unsigned mx = rfl(umax(umax(umax(c0.y, c0.z), umax(c0.w, c1.x)), umax(umax(c1.y, c1.z), c1.w)));
          if (mx + PLEAD >= u && mn + PRING > u + 1u) break;
          __builtin_amdgcn_s_sleep(4);
        }
        asm volatile("s_waitcnt vmcnt(0)" : "+v"(tk) :: "memory");
        PCVT(D, L0)
#pragma unroll
        for (int i = 0; i < 32; ++i) asm volatile("" : "+v"(D[i]));
        const unsigned on = rfl(tk) - tbase + 2u * NPWG;
        PLOAD(L0, umin(on, NUNIT - 1))
        if (on < (unsigned)NUNIT) PTICKET(tk)
        SCHED();
        PPUB(D, u, o, kRaw || image_of(o) % NST >= (unsigned)(NST - RAWTAIL))
        if (on >= (unsigned)NUNIT) {
          lds_st(uidAddr + ((u + 1u) % PRING) * 8, 0xffffffffu);
          WAIT_L(0);
          lds_st(convAddr, u + 2u);
          break;
        }
        o = on;
      }
      STAMP(16);
    } else {
      const unsigned call = rfl(__hip_atomic_load(callp, __ATOMIC_RELAXED, __HIP_MEMORY_SCOPE_AGENT)) + 1u;
      if (lane == 0) __hip_atomic_store(callp, call, __ATOMIC_RELAXED, __HIP_MEMORY_SCOPE_AGENT);
      const unsigned dest = (unsigned)(rank + wave) & 7u;
      char* dws = p.ws[dest < (unsigned)NR ? dest : (unsigned)rank];
      const char* xd = dws + WS_X16 + (size_t)(call & 1u) * X16_PAR + (size_t)(rank * 4) * NST * A_IMG;
      const char* fd = dws + WS_FLAGA + (size_t)(call & 1u) * FLAGA_PAR + (size_t)(rank * 4) * NST * 8;
      const unsigned voff2 = voff + 4096;
      unsigned conv = 0, prev = 0, prevMeta = 0;
      for (unsigned u = 0;; ++u) {
        while (conv <= u) {
          conv = rfl(lds_ld_wait(convAddr));
          if (conv <= u) __builtin_amdgcn_s_sleep(1);
        }
        const unsigned o = rfl(lds_ld_wait(uidAddr + (u % PRING) * 8));
        if (o >= (unsigned)NUNIT) break;
        const unsigned meta = rfl(lds_ld_wait(uidAddr + (u % PRING) * 8 + 4));
        const unsigned nl = meta == META_RAW ? 64u : umax((((meta >> 8) + 31u) >> 5) << 2, 4u);
        const unsigned long long lastMask = nl == 64u ? ~0ull : (1ull << nl) - 1ull;
        const unsigned la = lds0 + (u % PRING) * A_IMG + voff;
        u4 d[8];
#pragma unroll
        for (int f = 0; f < 8; ++f) d[f] = *(const LAS u4*)(size_t)(la + f * 1024);
        asm volatile("s_waitcnt lgkmcnt(0)" : "+v"(d[0]), "+v"(d[1]), "+v"(d[2]), "+v"(d[3]), "+v"(d[4]), "+v"(d[5]),
                     "+v"(d[6]), "+v"(d[7]) :: "memory");
        lds_st(sentAddr + wave * 4, u + 1u);
        const unsigned im = image_of(o);
        const char* db = xd + (size_t)im * A_IMG;
#define SEND_LAST()                                                                          \
        {                                                                                    \
          unsigned long long keep;                                                           \
          asm volatile("s_mov_b64 %0, exec\n\ts_mov_b64 exec, %4\n\ts_nop 2\n\t"             \
                       "global_store_dwordx4 %1, %2, %3 offset:3072 sc0 sc1\n\ts_nop 1\n\ts_mov_b64 exec, %0" \
                       : "=&s"(keep) : "v"(voff2), "v"(d[7]), "s"(db), "s"(lastMask) : "memory"); \
        }
        if (o < (unsigned)(NUNIT - OPENWIN)) {
#pragma unroll
          for (int f = 0; f < 8; ++f) {
            WAIT_V(PWIN - 1);
            if (f == PWIN - 1 && u > 0)
              flag_store2(fd + (size_t)prev * 8, 0u, call, prevMeta);
            if (f < 7) STORE16_WT(f < 4 ? voff : voff2, d[f], db, (f & 3) * 1024);
            else SEND_LAST()
          }
        } else {
#pragma unroll
          for (int f = 0; f < 7; ++f) STORE16_WT(f < 4 ? voff : voff2, d[f], db, (f & 3) * 1024);
          SEND_LAST()
          WAIT_V(8);
          flag_store2(fd + (size_t)prev * 8, 0u, call, prevMeta);
        }
        prev = im;
        prevMeta = meta;
      }
      WAIT_V(0);
      flag_store2(fd + (size_t)prev * 8, 0u, call, prevMeta);
      STAMP(16 + wave);
    }
    return;
  }

  const int gl = (xcd >> 1) * 29 + (idx - PPX);
  const int tr = gl >> 2;
  const int cb = (xcd & 1) * 4 + (gl & 3);
  const int gidx = (idx - PPX) * 8 + xcd;
  const int blk = tr < 28 ? (((rank + 1 + (tr >> 2)) & 7) * 4 + (tr & 3)) : rank * 4 + 3;
  const int lfrag = tr < 28 ? tr : 24;
  const unsigned fullAddr = lds0 + CTRL;
  const unsigned freeAddr = lds0 + CTRL + 16;
  const unsigned badAddr = lds0 + CTRL + 32;

  const unsigned long long tstart = __builtin_amdgcn_s_memrealtime();
  const unsigned call = rfl(__hip_atomic_load(callp, __ATOMIC_RELAXED, __HIP_MEMORY_SCOPE_AGENT)) + 1u;
  if (lane == 0) __hip_atomic_store(callp, call, __ATOMIC_RELAXED, __HIP_MEMORY_SCOPE_AGENT);

  if (wave < 4) {
    __builtin_amdgcn_s_setprio(CPRIO);
    const unsigned arow = lane & 15, ako = lane >> 4;
    const unsigned offA = lds0 + arow * 64 + ((ako ^ ((arow >> 3) << 1)) << 4);
    const unsigned myFree = freeAddr + wave * 4;
    u4 wrs;
    {
      const unsigned long long wb = (unsigned long long)(myws + WS_W16 + (size_t)cb * NST * B_IMG);
      wrs.x = rfl((unsigned)wb);
      wrs.y = rfl((unsigned)(wb >> 32)) & 0xffffu;
      wrs.z = NST * B_IMG;
      wrs.w = 0x00020000u;
    }
    const unsigned voffB = wave * 4096 + lane * 16;

    f4 acc[9][4];
#pragma unroll
    for (int i = 0; i < 9; ++i)
#pragma unroll
      for (int j = 0; j < 4; ++j) {
        acc[i][j] = (f4){0.f, 0.f, 0.f, 0.f};
        asm volatile("" : "+v"(acc[i][j]));
      }
    u4 a[6], b[4][4];
#pragma unroll
    for (int i = 0; i < 6; ++i) a[i] = (u4){0u, 0u, 0u, 0u};
#pragma unroll
    for (int i = 0; i < 16; ++i) b[i >> 2][i & 3] = (u4){0u, 0u, 0u, 0u};

#define LDA(dst, addr, imm) asm volatile("ds_read_b128 %0, %1 offset:%2" : "+v"(dst) : "v"(addr), "i"(imm) : "memory")
#define LDB(dst, soff, j) asm volatile("buffer_load_dwordx4 %0, %1, %2, %3 offen offset:%4 sc0 sc1" \
                                       : "+v"(dst) : "v"(voffB), "s"(wrs), "s"(soff), "i"((j) * 1024) : "memory")
#define MFMA(c, av, bv) asm volatile("v_mfma_f32_16x16x32_f16 %0, %1, %2, %0" : "+v"(c) : "v"(av), "v"(bv))
    unsigned ready = 0;
    auto wait_ready = [&](unsigned need) {
      while (ready < need) {
        const u4 f = lds_ld4_wait(fullAddr);
        ready = rfl(umin(umin(f.x, f.y), f.z));
        if (ready < need) __builtin_amdgcn_s_sleep(1);
      }
    };
    wait_ready(1);
    if (wave == 0) STAMP(2);
    asm volatile("s_waitcnt vmcnt(0) lgkmcnt(0)\n\ts_nop 4" ::: "memory");
#pragma unroll
    for (int t = 0; t < 3; ++t)
#pragma unroll
      for (int j = 0; j < 4; ++j) LDB(b[t][j], t * B_IMG, j);
#pragma unroll
    for (int i = 0; i < 6; ++i) LDA(a[i], offA, i * 1024);
    asm volatile("s_waitcnt vmcnt(8) lgkmcnt(0)" ::: "memory");

    unsigned vA = offA;
    unsigned vFull = fullAddr, vFree = myFree;
    u4 fw = {0u, 0u, 0u, 0u};
    unsigned fm = 0;
    asm volatile("" : "+v"(vFull), "+v"(vFree));
    for (unsigned s4 = 0; s4 < (unsigned)NST; s4 += 4) {
      unsigned vN, soffB;
#pragma unroll
      for (int t = 0; t < 4; ++t) {
#pragma unroll
        for (int i = 0; i < 9; ++i) {
          const int buf = (t * 9 + i) % 6;
#pragma unroll
          for (int j = 0; j < 4; ++j) {
            MFMA(acc[i][j], a[buf], b[t][j]);
            SCHED();
            if (j == 3) {
              if (i < 3) LDA(a[buf], vA, t * ST + (i + 6) * 1024);
              else if (t < 3) LDA(a[buf], vA, (t + 1) * ST + (i - 3) * 1024);
              else LDA(a[buf], vN, (i - 3) * 1024);
            }
            if (i == 0 && j == 0) { soffB = (s4 + t + 3u) * B_IMG; asm volatile("" : "+s"(soffB)); }
            if (i == 0 && j == 1) asm volatile("ds_read_b128 %0, %1" : "+v"(fw) : "v"(vFull) : "memory");
            if (i == 1 && j == 0 && t == 3) {
              vN = offA + ((s4 + 4u) & (NSLOT - 1)) * ST;
              asm volatile("" : "+v"(vN));
            }
            if (i == 2 && j == 2) asm volatile("s_waitcnt lgkmcnt(2)" : "+v"(fw) :: "memory");
            if (i == 2 && j == 3) { fm = umin(umin(fw.x, fw.y), fw.z); asm volatile("" : "+v"(fm)); }
            if (i == 3 && j == 0) { ready = rfl(fm); asm volatile("" : "+s"(ready)); }
            if (i == 3 && j == 2) { if (ready < s4 + t + 2u) wait_ready(s4 + t + 2u); }
            if (i >= BPOS && i < BPOS + 4 && j == (i == 0 ? 2 : 1)) LDB(b[(t + 3) & 3][i - BPOS], soffB, i - BPOS);
            if (i == 5 && j == 2) {
              WAIT_L(2);
              if (t == 3) lds_st(vFree, s4 + 4u);
            }
            if (i == 8 && j == 2) asm volatile("s_waitcnt vmcnt(8) lgkmcnt(2)" ::: "memory");
            SCHED();
          }
        }
      }
      vA = vN;
    }
    asm volatile("s_waitcnt vmcnt(0) lgkmcnt(0)" ::: "memory");
    asm volatile("s_nop 7\n\ts_nop 7" ::: "memory");
    SCHED();
    if (wave == 0) STAMP(3);

    const unsigned bad = rfl(lds_ld_wait(badAddr));
    float* ocol = p.out + (size_t)((lane >> 4) * 4) * N_SH + cb * BN + wave * 64 + (lane & 15) * 4;
    const float nanv = __builtin_nanf("");
#pragma unroll
    for (int i = 0; i < 9; ++i) {
      if (tr == 28 && (i < 4 || i == 8)) continue;
      float* orow = ocol + (size_t)(i < 8 ? blk * BM + i * 16 : rank * M_SH + lfrag * 16) * N_SH;
#pragma unroll
      for (int r = 0; r < 4; ++r) {
        f4 o = {acc[i][0][r], acc[i][1][r], acc[i][2][r], acc[i][3][r]};
        if (bad) o = (f4){nanv, nanv, nanv, nanv};
        asm volatile("global_store_dwordx4 %0, %1, off" EPI_BITS "\n\ts_nop 1" ::"v"(orow + (size_t)r * N_SH), "v"(o) : "memory");
      }
    }
  } else if (wave < 7) {
    __builtin_amdgcn_s_setprio(FPRIO);
    const unsigned fd = wave - 4;
    const char* x16 = myws + WS_X16 + (size_t)(call & 1u) * X16_PAR;
    const char* ga = x16 + (size_t)blk * NST * A_IMG;
    const char* gb = x16 + (size_t)(rank * 4 + (lfrag >> 3)) * NST * A_IMG;
    const char* fa = myws + WS_FLAGA + (size_t)(call & 1u) * FLAGA_PAR + (size_t)blk * NST * 8;
    const unsigned* fl = (const unsigned*)(myws + WS_FLAGL) + (lfrag >> 3) * NST;
    const unsigned* fw = (const unsigned*)(myws + WS_FLAGW) + cb * NST;
    unsigned* const tmo = (unsigned*)(myws + WS_TMO);
    const unsigned voff = lane * 16, voffe = EXC_OFF + lane * 2, voffl = (lfrag & 7) * 1024 + voff;
    const unsigned pl = 3u * umin(lane, 7u);
    const unsigned myFull = fullAddr + fd * 4;
    lds_st(myFull, fd);
    bool allRaw = tr == 28;
    unsigned arrA = allRaw ? 2u * NST : 0u;
    unsigned pbase = 0;
    unsigned arrL = 0;
    unsigned freed = 0;
    unsigned pmeta = 0;
    u4 RA[9], RB[9];
    unsigned eA0, eA1, eB0, eB1, metaA = 0, metaB = 0, tvA = 0, tvB = 0;
#define POLL_LOAD(sf) __hip_atomic_load((const unsigned long long*)(fa + umin((sf) + pl, NST - 1) * 8u), __ATOMIC_RELAXED, __HIP_MEMORY_SCOPE_SYSTEM)
#define POLL_TAKE(pn, sf)                                                                    \
    {                                                                                        \
      const unsigned ok = (unsigned)__ballot((unsigned)(pn) >= call) & 0xffu;                \
      arrA = (sf) + 3u * (unsigned)__builtin_ctz(~ok);                                       \
      pbase = (sf);                                                                          \
      pmeta = (unsigned)((pn) >> 32);                                                        \
    }
#define FETCH(R, e0, e1, meta, tvn, sf, real)                                                     \
    {                                                                                        \
      meta = allRaw ? META_RAW : (unsigned)__builtin_amdgcn_readlane(pmeta, (((sf) - pbase) / 3u) & 63u); \
      const unsigned nrec = (real) ? (unsigned)(NST * A_IMG) : 0u;                           \
      const __amdgpu_buffer_rsrc_t ra = __builtin_amdgcn_make_buffer_rsrc((void*)ga, 0, nrec, 0x00020000); \
      const __amdgpu_buffer_rsrc_t rb = __builtin_amdgcn_make_buffer_rsrc((void*)gb, 0, nrec, 0x00020000); \
      const unsigned so = ((sf) & (NST - 1)) * A_IMG;                                        \
      tvn = __hip_atomic_load(tmo, __ATOMIC_RELAXED, __HIP_MEMORY_SCOPE_AGENT);              \
      _Pragma("unroll") for (int f = 0; f < 4; ++f) {                                        \
        R[4 + f] = __builtin_amdgcn_raw_buffer_load_b128(ra, voff + f * 1024, so + 4096, 17); \
        R[f] = __builtin_amdgcn_raw_buffer_load_b128(ra, voff + f * 1024, so, 17);           \
      }                                                                                      \
      e0 = __builtin_amdgcn_raw_buffer_load_b16(ra, voffe, so, 17);                          \
      e1 = __builtin_amdgcn_raw_buffer_load_b16(ra, voffe + 128, so, 17);                    \
      R[8] = __builtin_amdgcn_raw_buffer_load_b128(rb, voffl, so, 17);                       \
      CFENCE();                                                                              \
    }
    auto await = [&](unsigned sf) {
      while (arrL < umin(sf + 1u + LOOK, NST)) {
        const unsigned fi = umin(arrL + lane, NST - 1);
        arrL += poll_flags(fl + fi, fw + fi, call, myws, badAddr, lane);
      }
      if (sf >= arrA) {
        const unsigned long long t0 = __builtin_amdgcn_s_memrealtime();
        for (unsigned spin = 1;; ++spin) {
          const unsigned long long pn = POLL_LOAD(sf);
          POLL_TAKE(pn, sf)
          if (sf < arrA) break;
          if ((spin & 63u) == 0u &&
              (rfl(__hip_atomic_load(tmo, __ATOMIC_RELAXED, __HIP_MEMORY_SCOPE_AGENT)) == call ||
               __builtin_amdgcn_s_memrealtime() - t0 > DEADLINE_TICKS)) {
            if (lane == 0) __hip_atomic_store(tmo, call, __ATOMIC_RELAXED, __HIP_MEMORY_SCOPE_AGENT);
            lds_st(badAddr, 1u);
            allRaw = true;
            arrA = 2u * NST;
            break;
          }
          __builtin_amdgcn_s_sleep(POLL_SLEEP);
        }
      }
      CFENCE();
    };
#define FEED(R, e0, e1, meta, tv, Rn, en0, en1, metan, tvn)                                           \
    {                                                                                        \
      const unsigned sn = s + 3u;                                                            \
      const bool more = sn < (unsigned)NST;                                                  \
      const bool early = more && sn < arrA && arrL >= umin(sn + 1u + LOOK, NST);             \
      const unsigned sp = early ? sn + 3u : sn;          \
      FETCH(Rn, en0, en1, metan, tvn, sn, early)                                                  \
      const unsigned long long pn = POLL_LOAD(sp);                                           \
      while (freed + NSLOT <= s) {                     \
        const u4 f = lds_ld4_wait(freeAddr);                                                 \
        freed = rfl(umin(umin(f.x, f.y), umin(f.z, f.w)));                                   \
        if (freed + NSLOT <= s) __builtin_amdgcn_s_sleep(2);                                 \
      }                                                                                      \
      const unsigned slot = lds0 + (s % NSLOT) * ST;                                         \
      if (meta != META_RAW) {                                                                \
        decode_image(R, e0, e1, slot, meta, lane);                                           \
      } else {                                                                               \
        _Pragma("unroll") for (int f = 0; f < 8; ++f)                                        \
          asm volatile("ds_write_b128 %0, %1 offset:%2" ::"v"(slot + voff), "v"(R[f]), "i"(f * 1024) : "memory"); \
      }                                                                                      \
      asm volatile("ds_write_b128 %0, %1 offset:8192" ::"v"(slot + voff), "v"(R[8]) : "memory"); \
      if (!more) {                                     \
        if (rfl(tv) == call) lds_st(badAddr, 1u);                                                 \
        WAIT_L(0);                                                                           \
        lds_st(myFull, 2u * NST);                                                            \
        break;                                                                               \
      }                                                                                      \
      WAIT_L(0);                                                                             \
      lds_st(myFull, sn);                                                                    \
      NG_STAMP()                                                                             \
      if (sp >= arrA) POLL_TAKE(pn, sp)                                                      \
      if (!early) {                                                                          \
        await(sn);                                                                           \
        FETCH(Rn, en0, en1, metan, tvn, sn, true)                                                 \
      }                                                                                      \
      s = sn;                                                                                \
    }
#define NG_STAMP()
    unsigned s = fd;
    await(s);
    FETCH(RA, eA0, eA1, metaA, tvA, s, true)
    for (;;) {
      FEED(RA, eA0, eA1, metaA, tvA, RB, eB0, eB1, metaB, tvB)
      FEED(RB, eB0, eB1, metaB, tvB, RA, eA0, eA1, metaA, tvA)
    }
    STAMP(wave + 1);
  } else {
    const unsigned voff = lane * 16;
    for (int item = gidx; item < NITEM; item += NGWG) {
      while (__builtin_amdgcn_s_memrealtime() - tstart < (unsigned long long)(item * CONV_TICKS)) __builtin_amdgcn_s_sleep(8);
      const int pair = item / 20, r = item % 20;
      if (r < 16) {
        const int wcb = r & 7, wks = pair * 2 + (r >> 3);
        const float* wsrc = p.w + (size_t)(wks * BK + (lane >> 4) * 8) * N_TOT + (size_t)rank * N_SH + wcb * BN + (lane & 15) * 4;
        f4 v[4][8];
#pragma unroll
        for (int g = 0; g < 4; ++g)
#pragma unroll
          for (int t = 0; t < 8; ++t) v[g][t] = *(const f4*)(wsrc + (size_t)t * N_TOT + g * 64);
        SCHED();
        const char* dst = myws + WS_W16 + ((size_t)wcb * NST + wks) * B_IMG;
#pragma unroll
        for (int g = 0; g < 4; ++g) {
          const char* dg = dst + g * 4096;
#pragma unroll
          for (int j = 0; j < 4; ++j) {
            u4 o;
            o.x = cvt2(v[g][0][j], v[g][1][j]);
            o.y = cvt2(v[g][2][j], v[g][3][j]);
            o.z = cvt2(v[g][4][j], v[g][5][j]);
            o.w = cvt2(v[g][6][j], v[g][7][j]);
            STORE16_WT(voff, o, dg, j * 1024);
          }
        }
        WAIT_V(0);
        flag_store(myws + WS_FLAGW + (size_t)(wcb * NST + wks) * 4, 0u, call);
      } else {
        const int rb = r - 16, ks = pair * 2;
        const float* xsrc = p.x + (size_t)(rb * BM + prow) * KD + ks * BK + pko * 8;
        f4 v[2][8][2];
#pragma unroll
        for (int f = 0; f < 8; ++f)
#pragma unroll
          for (int h = 0; h < 2; ++h) {
            v[h][f][0] = *(const f4*)(xsrc + (size_t)f * 16 * KD + h * BK);
            v[h][f][1] = *(const f4*)(xsrc + (size_t)f * 16 * KD + h * BK + 4);
          }
        SCHED();
        const char* dst = myws + WS_X16 + (size_t)(call & 1u) * X16_PAR + ((size_t)(rank * 4 + rb) * NST + ks) * A_IMG;
#pragma unroll
        for (int h = 0; h < 2; ++h)
#pragma unroll
          for (int f = 0; f < 8; ++f) {
            u4 o;
            o.x = cvt2(v[h][f][0][0], v[h][f][0][1]);
            o.y = cvt2(v[h][f][0][2], v[h][f][0][3]);
            o.z = cvt2(v[h][f][1][0], v[h][f][1][1]);
            o.w = cvt2(v[h][f][1][2], v[h][f][1][3]);
            const char* dg = dst + h * A_IMG + (f >> 2) * 4096;
            STORE16_WT(voff, o, dg, (f & 3) * 1024);
          }
        WAIT_V(0);
        flag_store2(myws + WS_FLAGL + (size_t)(rb * NST + ks) * 4, 0u, call, call);
      }
    }
    STAMP(1);
  }
}

}

extern "C" void kernel_launch(int rank, int nranks, ncclComm_t comm,
                              hipStream_t stream,
                              void* const* d_in, const int* in_sizes, int n_in,
                              void* d_out, int out_size,
                              const int* in_dtypes, int out_dtype,
                              void* const* d_out_all,
                              void* const* d_ws_all, size_t ws_bytes) {
  (void)comm; (void)in_dtypes; (void)out_dtype; (void)d_out_all;
  if (nranks != NR || (unsigned)rank >= (unsigned)NR || n_in != 2 || ws_bytes < WS_END ||
      in_sizes[0] != M_SH * KD || in_sizes[1] != KD * N_TOT || out_size != NR * M_SH * N_SH) {
    fprintf(stderr, "kernel_launch: unexpected problem shape\n");
    return;
  }
  static bool attr_set[NR] = {false, false, false, false, false, false, false, false};
  if (!attr_set[rank]) {
    HIP_CHECK(hipFuncSetAttribute((const void*)fused_kernel, hipFuncAttributeMaxDynamicSharedMemorySize, LDS_BYTES));
    attr_set[rank] = true;
  }
  Params p;
  p.x = (const float*)d_in[0];
  p.w = (const float*)d_in[1];
  p.out = (float*)d_out;
  for (int i = 0; i < NR; ++i) p.ws[i] = (char*)d_ws_all[i];
  p.rank = rank;
  p.pad = 0;
  hipLaunchKernelGGL(fused_kernel, dim3(NBLK), dim3(512), LDS_BYTES, stream, p);
  HIP_CHECK(hipGetLastError());
}
```

```cpp
#include <hip/hip_runtime.h>
#include <rccl/rccl.h>
#include <cstdio>

#define HIP_CHECK(expr) do { hipError_t _e = (expr); if (_e != hipSuccess) { \
    fprintf(stderr, "HIP error %s at %s:%d\n", hipGetErrorString(_e), __FILE__, __LINE__); \
    return; } } while (0)

#define LAS __attribute__((address_space(3)))
typedef _Float16 h8 __attribute__((ext_vector_type(8)));
typedef _Float16 h2 __attribute__((ext_vector_type(2)));
typedef float f4 __attribute__((ext_vector_type(4)));
typedef float f2 __attribute__((ext_vector_type(2)));
typedef unsigned u4 __attribute__((ext_vector_type(4)));
typedef unsigned u2 __attribute__((ext_vector_type(2)));
typedef unsigned short us2 __attribute__((ext_vector_type(2)));

namespace {

constexpr int NR = 8;
constexpr int M_SH = 512, KD = 4096, N_SH = 2048, N_TOT = 16384;
constexpr int BM = 128, BN = 256, BK = 32;
constexpr int NST = KD / BK;
constexpr int A_IMG = BM * BK * 2;
constexpr int B_IMG = BN * BK * 2;
constexpr int ST = A_IMG + 1024;
constexpr int NSLOT = 16;
constexpr int CTRL = NSLOT * ST;
constexpr int LDS_BYTES = CTRL + 1024;
constexpr int NBLK = 256;
#define PPX 3
#define NPACT (8 * PPX)
constexpr int NPWG = NPACT;
constexpr int NGWG = 232;
constexpr int NITEM = 1280;
constexpr int NUNIT = 4 * NST;
constexpr int PRING = 16;
#define PWIN 3
#define PLEAD 0u
#define BPOS 0
#define LEAD0 8
#define LEAD1 0
#define OPENWIN 0
#define POLL_SLEEP 1
#define RAWTAIL 2
#define CPRIO 2
#define FPRIO 1
#define EPI_BITS " sc0 sc1"
#define XP_AUX 0
#define XL_LD(p) (*(p))
#define CONV_TICKS 3
constexpr int LOOK = 3;

constexpr size_t WS_CALL = 0;
constexpr size_t WS_TMO = 32768;
constexpr size_t WS_TICKET = 32768 + 256;
constexpr size_t WS_FLAGA = 65536;
constexpr size_t FLAGA_PAR = (size_t)NR * 4 * NST * 8;
constexpr size_t WS_FLAGW = WS_FLAGA + 2 * FLAGA_PAR;
constexpr size_t WS_FLAGL = WS_FLAGW + 8 * NST * 4;
constexpr size_t WS_DBG = 262144;
constexpr size_t WS_X16 = 1u << 20;
constexpr size_t X16_PAR = (size_t)NR * 4 * NST * A_IMG;
constexpr size_t WS_W16 = WS_X16 + 2 * X16_PAR;
constexpr size_t WS_XL = WS_W16 + (size_t)8 * NST * B_IMG;
constexpr size_t WS_END = WS_XL + (size_t)4 * NST * A_IMG;

constexpr unsigned META_RAW = 0x80000000u;
constexpr int EXC_OFF = 7168;
constexpr unsigned MAXEXC = 128;

constexpr unsigned long long DEADLINE_TICKS = 40000000ull;

struct Params {
  const float* x;
  const float* w;
  float* out;
  char* ws[NR];
  int rank;
  int pad;
};

#define SCHED() __builtin_amdgcn_sched_barrier(0)
#define CFENCE() asm volatile("" ::: "memory")
#define WAIT_V(n) asm volatile("s_waitcnt vmcnt(%0)" ::"n"(n) : "memory")
#define WAIT_L(n) asm volatile("s_waitcnt lgkmcnt(%0)" ::"n"(n) : "memory")

__device__ __forceinline__ unsigned rfl(unsigned v) { return __builtin_amdgcn_readfirstlane(v); }
__device__ __forceinline__ unsigned umin(unsigned a, unsigned b) { return a < b ? a : b; }
__device__ __forceinline__ unsigned umax(unsigned a, unsigned b) { return a > b ? a : b; }

__device__ __forceinline__ unsigned cvt2(float a, float b) {
  f2 v = {a, b};
  h2 h = __builtin_convertvector(v, h2);
  return __builtin_bit_cast(unsigned, h);
}

__device__ __forceinline__ unsigned image_of(unsigned o) {
  constexpr unsigned MID = NST - LEAD0 - LEAD1;
  if (o < LEAD0) return o;
  o -= LEAD0;
  if (o < 2u * LEAD1) return (o & 1u) ? NST + (o >> 1) : LEAD0 + (o >> 1);
  o -= 2u * LEAD1;
  if (o < 4u * MID) {
    const unsigned rb = o & 3u;
    return rb * NST + (o >> 2) + (rb == 0u ? LEAD0 + LEAD1 : rb == 1u ? LEAD1 : 0u);
  }
  o -= 4u * MID;
  if (o < 3u * LEAD0) {
    const unsigned rb = 1u + o % 3u;
    return rb * NST + MID + o / 3u + (rb == 1u ? LEAD1 : 0u);
  }
  o -= 3u * LEAD0;
  return (2u + (o & 1u)) * NST + NST - LEAD1 + (o >> 1);
}

__device__ __forceinline__ void lds_st(unsigned addr, unsigned v) {
  asm volatile("ds_write_b32 %0, %1" ::"v"(addr), "v"(v) : "memory");
}
__device__ __forceinline__ u4 lds_ld4_wait(unsigned addr) {
  u4 r;
  asm volatile("ds_read_b128 %0, %1\n\ts_waitcnt lgkmcnt(0)" : "=&v"(r) : "v"(addr) : "memory");
  return r;
}
__device__ __forceinline__ unsigned lds_ld_wait(unsigned addr) {
  unsigned r;
  asm volatile("ds_read_b32 %0, %1\n\ts_waitcnt lgkmcnt(0)" : "=&v"(r) : "v"(addr) : "memory");
  return r;
}
__device__ __forceinline__ void flag_store(const char* sbase, unsigned voff, unsigned val) {
  unsigned long long keep;
  asm volatile(
      "s_mov_b64 %0, exec\n\t"
      "s_mov_b64 exec, 1\n\t"
      "s_nop 2\n\t"
      "global_store_dword %1, %2, %3 sc0 sc1\n\t"
      "s_mov_b64 exec, %0"
      : "=&s"(keep)
      : "v"(voff), "v"(val), "s"(sbase)
      : "memory");
}
__device__ __forceinline__ void flag_store2(const char* sbase, unsigned voff, unsigned lo, unsigned hi) {
  unsigned long long keep;
  const u2 v = {lo, hi};
  asm volatile(
      "s_mov_b64 %0, exec\n\t"
      "s_mov_b64 exec, 1\n\t"
      "s_nop 2\n\t"
      "global_store_dwordx2 %1, %2, %3 sc0 sc1\n\t"
      "s_mov_b64 exec, %0"
      : "=&s"(keep)
      : "v"(voff), "v"(v), "s"(sbase)
      : "memory");
}
__device__ __forceinline__ unsigned perm(unsigned hi, unsigned lo, unsigned sel) { return __builtin_amdgcn_perm(hi, lo, sel); }
__device__ __forceinline__ unsigned xad(unsigned a, unsigned b, unsigned c) {
  unsigned r;
  asm("v_xad_u32 %0, %1, %2, %3" : "=v"(r) : "v"(a), "v"(b), "s"(c));
  return r;
}

__device__ __forceinline__ unsigned pack_image(const unsigned (&D)[32], unsigned slot, int lane, bool force_raw,
                                               unsigned& base) {
  unsigned meta = META_RAW;
  for (int attempt = 0; attempt < 2 && !force_raw; ++attempt) {
    const unsigned bb = base * 0x04040404u;
    unsigned T[16], Q[4], c = 0;
#pragma unroll
    for (int g = 0; g < 4; ++g) {
      unsigned dl[4];
#pragma unroll
      for (int k = 0; k < 4; ++k) {
        const unsigned H = perm(D[8 * g + 2 * k + 1], D[8 * g + 2 * k], 0x07050301u);
        const unsigned R = (H | 0x80808080u) - bb;
        T[4 * g + k] = (R & 0x1f1f1f1fu) | (H & 0x80808080u);
        dl[k] = H ^ (T[4 * g + k] + bb);
      }
      Q[g] = (dl[0] >> 5) | (dl[1] >> 3) | (dl[2] >> 1) | (dl[3] << 1);
      c += (unsigned)__builtin_popcount((Q[g] | (Q[g] >> 1)) & 0x55555555u);
    }
    unsigned cnt = 0, ix = 0;
#pragma unroll
    for (int bt = 0; bt < 3; ++bt) {
      const unsigned long long m = __ballot((c >> bt) & 1u);
      ix += __builtin_amdgcn_mbcnt_hi((unsigned)(m >> 32), __builtin_amdgcn_mbcnt_lo((unsigned)m, 0u)) << bt;
      cnt += (unsigned)__builtin_popcountll(m) << bt;
    }
    if (__ballot(c > 7u) != 0ull) cnt = MAXEXC + 1;
    if (cnt > 96u && attempt == 0) {
      us2 mx = {0, 0};
      unsigned am = 0x7fff7fffu;
      asm volatile("" : "+s"(am));
#pragma unroll
      for (int i = 0; i < 32; ++i) mx = __builtin_elementwise_max(mx, __builtin_bit_cast(us2, D[i] & am));
      const unsigned el = (unsigned)(mx.x > mx.y ? mx.x : mx.y) >> 10;
      unsigned lo = 7, hi = 31;
      while (lo < hi) {
        const unsigned mid = (lo + hi) >> 1;
        if (__builtin_popcountll(__ballot(el > mid)) <= 20) hi = mid; else lo = mid + 1;
      }
      base = lo - 7;
      continue;
    }
    if (cnt > MAXEXC) break;
    meta = base | (cnt << 8);
    unsigned ea = slot + EXC_OFF + ix * 2u;
#pragma unroll
    for (int g = 0; g < 4; ++g) {
      unsigned q = Q[g];
      while (q != 0u) {
        const unsigned pos = (unsigned)__builtin_ctz(q) & ~1u;
        const unsigned dv = (q >> pos) & 3u;
        q &= ~(3u << pos);
        const unsigned hw = (4u * g + ((pos >> 1) & 3u)) * 256u + (unsigned)lane * 4u + (pos >> 3);
        *(LAS unsigned short*)(size_t)ea = (unsigned short)(hw | (dv << 12));
        ea += 2u;
      }
    }
    const unsigned la = slot + lane * 16;
    unsigned P[12];
#pragma unroll
    for (int g = 0; g < 4; ++g) {
      u4 Lp;
#pragma unroll
      for (int k = 0; k < 4; ++k) Lp[k] = perm(D[8 * g + 2 * k + 1], D[8 * g + 2 * k], 0x06040200u);
      *(LAS u4*)(size_t)(la + g * 1024) = Lp;
      const unsigned t3 = T[4 * g + 3];
      P[3 * g + 0] = T[4 * g + 0] | ((t3 << 5) & 0x60606060u);
      P[3 * g + 1] = T[4 * g + 1] | ((t3 << 3) & 0x60606060u);
      P[3 * g + 2] = T[4 * g + 2] | ((t3 << 1) & 0x20202020u) | ((t3 >> 1) & 0x40404040u);
    }
#pragma unroll
    for (int t = 0; t < 3; ++t)
      *(LAS u4*)(size_t)(la + (4 + t) * 1024) = (u4){P[4 * t], P[4 * t + 1], P[4 * t + 2], P[4 * t + 3]};
    break;
  }
  if (meta == META_RAW) {
#pragma unroll
    for (int j = 0; j < 16; ++j) *(LAS u2*)(size_t)(slot + j * 512 + lane * 8) = (u2){D[2 * j], D[2 * j + 1]};
  }
  return meta;
}

__device__ __forceinline__ void unpack_image(const u4 (&R)[9], unsigned e0, unsigned e1, unsigned slot, unsigned meta,
                                             int lane) {
  const unsigned la = slot + lane * 8;
  const unsigned bb = (meta & 31u) * 0x04040404u;
  const unsigned cnt = meta >> 8;
  unsigned H[16], X[12], A[4], S[4];
#pragma unroll
  for (int i = 0; i < 12; ++i) X[i] = R[4 + (i >> 2)][i & 3] & 0x60606060u;
  SCHED();
#pragma unroll
  for (int i = 0; i < 12; ++i) H[(i / 3) * 4 + i % 3] = xad(R[4 + (i >> 2)][i & 3], X[i], bb);
  SCHED();
#pragma unroll
  for (int i = 0; i < 12; ++i) { X[i] >>= 5 - 2 * (i % 3); asm volatile("" : "+v"(X[i])); }
  SCHED();
  const unsigned e = (unsigned)lane < cnt ? e0 : 0u;
#pragma unroll
  for (int g = 0; g < 4; ++g) { A[g] = X[3 * g] | X[3 * g + 1] | X[3 * g + 2]; S[g] = X[3 * g + 2] & 0x20202020u; }
  SCHED();
#pragma unroll
  for (int g = 0; g < 4; ++g) A[g] = xad(A[g], S[g], bb);
  const unsigned xa = slot + ((e & 0xffeu) << 1), xm = (e >> 12) << (13u + ((e & 1u) << 4));
  SCHED();
#pragma unroll
  for (int g = 0; g < 4; ++g) H[4 * g + 3] = A[g] | (S[g] << 2);
  SCHED();
  u2 o[17];
#pragma unroll
  for (int j = 0; j < 17; ++j) {
    if (j < 16) {
      o[j] = (u2){perm(H[j], R[j >> 2][j & 3], 0x05010400u), perm(H[j], R[j >> 2][j & 3], 0x07030602u)};
      asm volatile("" : "+v"(o[j]));
    }
    if (j >= 1)
      asm volatile("ds_write_b64 %0, %1 offset:%2" ::"v"(la), "v"(o[j - 1]), "i"((j - 1) * 512), "v"(o[j >= 2 ? j - 2 : 0]) : "memory");
  }
  asm volatile("ds_xor_b32 %0, %1" ::"v"(xa), "v"(xm) : "memory");
  if (cnt > 64u) {
    const unsigned f = (unsigned)lane + 64u < cnt ? e1 : 0u;
    asm volatile("ds_xor_b32 %0, %1" ::"v"(slot + ((f & 0xffeu) << 1)), "v"((f >> 12) << (13u + ((f & 1u) << 4))) : "memory");
  }
}

__device__ __forceinline__ void touch(const void* addr, unsigned lds_scratch) {
  unsigned keep;
  asm volatile("s_mov_b32 %0, m0\n\ts_mov_b32 m0, %2\n\ts_nop 0\n\tglobal_load_lds_dword %1, off sc0 sc1\n\ts_mov_b32 m0, %0"
               : "=&s"(keep) : "v"(addr), "s"(lds_scratch) : "memory");
}

#define STORE16_WT(voff, data, sbase, imm)                                                   \
  asm volatile("global_store_dwordx4 %0, %1, %2 offset:%3 sc0 sc1\n\ts_nop 1"                \
               ::"v"(voff), "v"(data), "s"(sbase), "i"(imm) : "memory")

#define STAMP(i) do {} while (0)

__device__ __forceinline__ unsigned poll_flags(const unsigned* f0, const unsigned* f1, unsigned call, char* myws,
                                               unsigned badAddr, int lane) {
  unsigned* const tmo = (unsigned*)(myws + WS_TMO);
  const unsigned long long t0 = __builtin_amdgcn_s_memrealtime();
  for (unsigned spin = 1;; ++spin) {
    const unsigned f = umin(__hip_atomic_load(f0, __ATOMIC_RELAXED, __HIP_MEMORY_SCOPE_SYSTEM),
                            __hip_atomic_load(f1, __ATOMIC_RELAXED, __HIP_MEMORY_SCOPE_SYSTEM));
    const unsigned long long ok = __ballot(f >= call);
    const unsigned n = ok == ~0ull ? 64u : (unsigned)__builtin_ctzll(~ok);
    if (n) return n;
    if ((spin & 63u) == 0u &&
        (rfl(__hip_atomic_load(tmo, __ATOMIC_RELAXED, __HIP_MEMORY_SCOPE_AGENT)) == call ||
         __builtin_amdgcn_s_memrealtime() - t0 > DEADLINE_TICKS)) {
      if (lane == 0) __hip_atomic_store(tmo, call, __ATOMIC_RELAXED, __HIP_MEMORY_SCOPE_AGENT);
      lds_st(badAddr, 1u);
      return 64u;
    }
    __builtin_amdgcn_s_sleep(2);
  }
}

__global__ __launch_bounds__(512, 2) void fused_kernel(Params p) {
  extern __shared__ __attribute__((aligned(1024))) char smem[];
  const unsigned lds0 = (unsigned)(unsigned long long)(LAS char*)smem;
  const int tid = threadIdx.x;
  const int lane = tid & 63;
  const int wave = (int)rfl((unsigned)(tid >> 6));
  const int bid = blockIdx.x;
  const int xcd = bid & 7, idx = bid >> 3;
  const int rank = p.rank;
  char* const myws = p.ws[rank];

  if (wave == 0) STAMP(0);

  unsigned* const callp = (unsigned*)(myws + WS_CALL + (size_t)bid * 128) + wave;

  const unsigned prow = lane >> 2;
  const unsigned pko = (lane & 3) ^ ((prow >> 3) << 1);

#define XLOAD(L, im)                                                                         \
  {                                                                                          \
    const float* sb = p.x + (size_t)((im) / NST * BM + prow) * KD + (im) % NST * BK + pko * 8; \
    _Pragma("unroll") for (int f = 0; f < 8; ++f) {                                          \
      L[2 * f] = XL_LD((const f4*)(sb + (size_t)f * 16 * KD));                               \
      L[2 * f + 1] = XL_LD((const f4*)(sb + (size_t)f * 16 * KD + 4));                       \
    }                                                                                        \
  }
#define XCVT(L, f) ((u4){cvt2(L[2 * (f)][0], L[2 * (f)][1]), cvt2(L[2 * (f)][2], L[2 * (f)][3]), \
                         cvt2(L[2 * (f) + 1][0], L[2 * (f) + 1][1]), cvt2(L[2 * (f) + 1][2], L[2 * (f) + 1][3])})
  constexpr bool kRaw = false;

  if (tid < 64) ((LAS unsigned*)(smem + CTRL))[tid] = 0u;
  WAIT_L(0);
  __builtin_amdgcn_s_barrier();
  CFENCE();

  if (idx < PPX) {
    if (bid >= NPWG) return;
    const unsigned convAddr = lds0 + CTRL;
    const unsigned sentAddr = lds0 + CTRL + 32;
    const unsigned uidAddr = lds0 + CTRL + 64;
    const unsigned voff = lane * 16;
    if (wave == 0) {
#define PPUB(u, o, meta)                                                                     \
      {                                                                                      \
        lds_st(uidAddr + ((u) % PRING) * 8, (o));                                            \
        lds_st(uidAddr + ((u) % PRING) * 8 + 4, (meta));                                     \
        WAIT_L(0);                                                                           \
        lds_st(convAddr, (u) + 1u);                                                          \
      }
      const __amdgpu_buffer_rsrc_t xr = __builtin_amdgcn_make_buffer_rsrc((void*)p.x, 0, M_SH * KD * 4, 0x00020000);
      const unsigned xo0 = (lane >> 3) * (KD * 4) + (lane & 7) * 16, xo1 = xo0 ^ 64u;
#define PLOAD(L, o)                                                                          \
      {                                                                                      \
        const unsigned im_ = image_of(o);                                                    \
        const unsigned sb = im_ / NST * (BM * KD * 4) + im_ % NST * (BK * 4);                \
        _Pragma("unroll") for (int j = 0; j < 16; ++j)                                       \
          L[j] = __builtin_bit_cast(f4, __builtin_amdgcn_raw_buffer_load_b128(xr, (j & 1) ? xo1 : xo0, sb + j * (8 * KD * 4), XP_AUX)); \
      }
#define PCVT(D, L)                                                                           \
      _Pragma("unroll") for (int j = 0; j < 16; ++j) {                                       \
        D[2 * j] = cvt2(L[j][0], L[j][1]);                                                   \
        D[2 * j + 1] = cvt2(L[j][2], L[j][3]);                                               \
        asm volatile("" : "+v"(D[2 * j]), "+v"(D[2 * j + 1]));                               \
      }
#define PTICKET(tk)                                                                          \
      {                                                                                      \
        unsigned long long keep;                                                             \
        asm volatile("s_mov_b64 %1, exec\n\ts_mov_b64 exec, 1\n\ts_nop 2\n\t"                \
                     "global_atomic_add %0, %2, %3, %4 sc0\n\ts_mov_b64 exec, %1"            \
                     : "=&v"(tk), "=&s"(keep) : "v"(0u), "v"(1u), "s"(ticket) : "memory");   \
      }
      f4 L0[16], Lx[16];
      unsigned D[32], base = 24;
      PLOAD(L0, (unsigned)bid)
      PLOAD(Lx, (unsigned)(bid + NPWG))
      const unsigned callv = __hip_atomic_load(callp, __ATOMIC_RELAXED, __HIP_MEMORY_SCOPE_AGENT);
      SCHED();
      PCVT(D, L0)
      pack_image(D, lds0, lane, true, base);
      PPUB(0u, (unsigned)bid, META_RAW)
      const unsigned call = rfl(callv) + 1u;
      unsigned* const ticket = (unsigned*)(myws + WS_TICKET);
      const unsigned tbase = (call - 1u) * 1024u;
      if (lane == 0) {
        __hip_atomic_store(callp, call, __ATOMIC_RELAXED, __HIP_MEMORY_SCOPE_AGENT);
        __hip_atomic_fetch_max(ticket, tbase, __ATOMIC_RELAXED, __HIP_MEMORY_SCOPE_AGENT);
      }
      WAIT_V(0);
      __builtin_amdgcn_s_setprio(1);
      unsigned o = bid + NPWG, tk;
      PTICKET(tk)
      for (unsigned u = 1;; ++u) {
        for (;;) {
          const u4 c0 = lds_ld4_wait(sentAddr), c1 = lds_ld4_wait(sentAddr + 16);
          const unsigned mn = rfl(umin(umin(umin(c0.y, c0.z), umin(c0.w, c1.x)), umin(umin(c1.y, c1.z), c1.w)));
          const unsigned mx = rfl(umax(umax(umax(c0.y, c0.z), umax(c0.w, c1.x)), umax(umax(c1.y, c1.z), c1.w)));
          if (mx + PLEAD >= u && mn + PRING > u + 1u) break;
          __builtin_amdgcn_s_sleep(2);
        }
        asm volatile("s_waitcnt vmcnt(0)" : "+v"(tk) :: "memory");
        __builtin_amdgcn_s_waitcnt(0x0f70);
        SCHED();
        PCVT(D, Lx)
        const unsigned on = rfl(tk) - tbase + 2u * NPWG;
        PLOAD(Lx, umin(on, NUNIT - 1))
        if (on < (unsigned)NUNIT) PTICKET(tk)
        SCHED();
        const unsigned meta = pack_image(D, lds0 + (u % PRING) * A_IMG, lane,
                                         kRaw || image_of(o) % NST >= (unsigned)(NST - RAWTAIL), base);
        PPUB(u, o, meta)
        if (on >= (unsigned)NUNIT) {
          PPUB(u + 1u, 0xffffffffu, 0u)
          break;
        }
        o = on;
      }
      STAMP(16);
    } else {
      const unsigned dest = (unsigned)(rank + wave) & 7u;
      char* dws = p.ws[dest < (unsigned)NR ? dest : (unsigned)rank];
      const unsigned call = rfl(__hip_atomic_load(callp, __ATOMIC_RELAXED, __HIP_MEMORY_SCOPE_AGENT)) + 1u;
      if (lane == 0) __hip_atomic_store(callp, call, __ATOMIC_RELAXED, __HIP_MEMORY_SCOPE_AGENT);
      const char* xd = dws + WS_X16 + (size_t)(call & 1u) * X16_PAR + (size_t)(rank * 4) * NST * A_IMG;
      const char* fd = dws + WS_FLAGA + (size_t)(call & 1u) * FLAGA_PAR + (size_t)(rank * 4) * NST * 8;
      const unsigned voff2 = voff + 4096;
      unsigned conv = 0, prev = 0, prevMeta = 0;
      for (unsigned u = 0;; ++u) {
        while (conv <= u) {
          conv = rfl(lds_ld_wait(convAddr));
          if (conv <= u) __builtin_amdgcn_s_sleep(1);
        }
        const unsigned o = rfl(lds_ld_wait(uidAddr + (u % PRING) * 8));
        if (o >= (unsigned)NUNIT) break;
        const unsigned meta = rfl(lds_ld_wait(uidAddr + (u % PRING) * 8 + 4));
        const unsigned nl = meta == META_RAW ? 64u : umax((((meta >> 8) + 31u) >> 5) << 2, 4u);
        const unsigned long long lastMask = nl == 64u ? ~0ull : (1ull << nl) - 1ull;
        const unsigned la = lds0 + (u % PRING) * A_IMG + voff;
        u4 d[8];
#pragma unroll
        for (int f = 0; f < 8; ++f) d[f] = *(const LAS u4*)(size_t)(la + f * 1024);
        asm volatile("s_waitcnt lgkmcnt(0)" : "+v"(d[0]), "+v"(d[1]), "+v"(d[2]), "+v"(d[3]), "+v"(d[4]), "+v"(d[5]),
                     "+v"(d[6]), "+v"(d[7]) :: "memory");
        lds_st(sentAddr + wave * 4, u + 1u);
        const unsigned im = image_of(o);
        const char* db = xd + (size_t)im * A_IMG;
#define SEND_LAST()                                                                          \
        {                                                                                    \
          unsigned long long keep;                                                           \
          asm volatile("s_mov_b64 %0, exec\n\ts_mov_b64 exec, %4\n\ts_nop 2\n\t"             \
                       "global_store_dwordx4 %1, %2, %3 offset:3072 sc0 sc1\n\ts_nop 1\n\ts_mov_b64 exec, %0" \
                       : "=&s"(keep) : "v"(voff2), "v"(d[7]), "s"(db), "s"(lastMask) : "memory"); \
        }
        if (o < (unsigned)(NUNIT - OPENWIN)) {
#pragma unroll
          for (int f = 0; f < 8; ++f) {
            WAIT_V(PWIN - 1);
            if (f == PWIN - 1 && u > 0)
              flag_store2(fd + (size_t)prev * 8, 0u, call, prevMeta);
            if (f < 7) STORE16_WT(f < 4 ? voff : voff2, d[f], db, (f & 3) * 1024);
            else SEND_LAST()
          }
        } else {
#pragma unroll
          for (int f = 0; f < 7; ++f) STORE16_WT(f < 4 ? voff : voff2, d[f], db, (f & 3) * 1024);
          SEND_LAST()
          WAIT_V(8);
          flag_store2(fd + (size_t)prev * 8, 0u, call, prevMeta);
        }
        prev = im;
        prevMeta = meta;
      }
      WAIT_V(0);
      flag_store2(fd + (size_t)prev * 8, 0u, call, prevMeta);
      STAMP(16 + wave);
    }
    return;
  }

  const int gl = (xcd >> 1) * 29 + (idx - PPX);
  const int tr = gl >> 2;
  const int cb = (xcd & 1) * 4 + (gl & 3);
  const int gidx = (idx - PPX) * 8 + xcd;
  const int blk = tr < 28 ? (((rank + 1 + (tr >> 2)) & 7) * 4 + (tr & 3)) : rank * 4 + 3;
  const int lfrag = tr < 28 ? tr : 24;
  const unsigned fullAddr = lds0 + CTRL;
  const unsigned freeAddr = lds0 + CTRL + 16;
  const unsigned badAddr = lds0 + CTRL + 32;

  const unsigned long long tstart = __builtin_amdgcn_s_memrealtime();
  const unsigned call = rfl(__hip_atomic_load(callp, __ATOMIC_RELAXED, __HIP_MEMORY_SCOPE_AGENT)) + 1u;
  if (lane == 0) __hip_atomic_store(callp, call, __ATOMIC_RELAXED, __HIP_MEMORY_SCOPE_AGENT);

  if (wave == 7) {
    const unsigned voff = lane * 16;
    for (int item = gidx; item < NITEM; item += NGWG) {
      while (__builtin_amdgcn_s_memrealtime() - tstart < (unsigned long long)(item * CONV_TICKS)) __builtin_amdgcn_s_sleep(8);
      const int pair = item / 20, r = item % 20;
      if (r < 16) {
        const int wcb = r & 7, wks = pair * 2 + (r >> 3);
        const float* wsrc = p.w + (size_t)(wks * BK + (lane >> 4) * 8) * N_TOT + (size_t)rank * N_SH + wcb * BN + (lane & 15) * 4;
        f4 v[4][8];
#pragma unroll
        for (int g = 0; g < 4; ++g)
#pragma unroll
          for (int t = 0; t < 8; ++t) v[g][t] = __builtin_nontemporal_load((const f4*)(wsrc + (size_t)t * N_TOT + g * 64));
        SCHED();
        const char* dst = myws + WS_W16 + ((size_t)wcb * NST + wks) * B_IMG;
#pragma unroll
        for (int g = 0; g < 4; ++g) {
          const char* dg = dst + g * 4096;
#pragma unroll
          for (int j = 0; j < 4; ++j) {
            u4 o;
            o.x = cvt2(v[g][0][j], v[g][1][j]);
            o.y = cvt2(v[g][2][j], v[g][3][j]);
            o.z = cvt2(v[g][4][j], v[g][5][j]);
            o.w = cvt2(v[g][6][j], v[g][7][j]);
            STORE16_WT(voff, o, dg, j * 1024);
          }
        }
        WAIT_V(0);
        flag_store(myws + WS_FLAGW + (size_t)(wcb * NST + wks) * 4, 0u, call);
      } else {
        const unsigned im = (r - 16) * NST + pair * 2;
        f4 L0[16], L1[16];
        XLOAD(L0, im)
        XLOAD(L1, im + 1)
        SCHED();
        const char* dst = myws + WS_XL + (size_t)im * A_IMG;
#pragma unroll
        for (int f = 0; f < 8; ++f) STORE16_WT(voff, XCVT(L0, f), dst + (f >> 2) * 4096, (f & 3) * 1024);
#pragma unroll
        for (int f = 0; f < 8; ++f) STORE16_WT(voff, XCVT(L1, f), dst + A_IMG + (f >> 2) * 4096, (f & 3) * 1024);
        WAIT_V(0);
        flag_store2(myws + WS_FLAGL + (size_t)im * 4, 0u, call, call);
      }
    }
    STAMP(1);
    return;
  }

  if (wave < 4) {
    __builtin_amdgcn_s_setprio(CPRIO);
    const unsigned arow = lane & 15, ako = lane >> 4;
    const unsigned offA = lds0 + arow * 64 + ((ako ^ ((arow >> 3) << 1)) << 4);
    const unsigned myFree = freeAddr + wave * 4;
    u4 wrs;
    {
      const unsigned long long wb = (unsigned long long)(myws + WS_W16 + (size_t)cb * NST * B_IMG);
      wrs.x = rfl((unsigned)wb);
      wrs.y = rfl((unsigned)(wb >> 32)) & 0xffffu;
      wrs.z = NST * B_IMG;
      wrs.w = 0x00020000u;
    }
    const unsigned voffB = wave * 4096 + lane * 16;

    f4 acc[9][4];
#pragma unroll
    for (int i = 0; i < 9; ++i)
#pragma unroll
      for (int j = 0; j < 4; ++j) {
        acc[i][j] = (f4){0.f, 0.f, 0.f, 0.f};
        asm volatile("" : "+v"(acc[i][j]));
      }
    u4 a[6], b[4][4];
#pragma unroll
    for (int i = 0; i < 6; ++i) a[i] = (u4){0u, 0u, 0u, 0u};
#pragma unroll
    for (int i = 0; i < 16; ++i) b[i >> 2][i & 3] = (u4){0u, 0u, 0u, 0u};

#define LDA(dst, addr, imm) asm volatile("ds_read_b128 %0, %1 offset:%2" : "+v"(dst) : "v"(addr), "i"(imm) : "memory")
#define LDB(dst, soff, j) asm volatile("buffer_load_dwordx4 %0, %1, %2, %3 offen offset:%4 sc0 sc1" \
                                       : "+v"(dst) : "v"(voffB), "s"(wrs), "s"(soff), "i"((j) * 1024) : "memory")
#define MFMA(c, av, bv) asm volatile("v_mfma_f32_16x16x32_f16 %0, %1, %2, %0" : "+v"(c) : "v"(av), "v"(bv))
    unsigned ready = 0;
    auto wait_ready = [&](unsigned need) {
      while (ready < need) {
        const u4 f = lds_ld4_wait(fullAddr);
        ready = rfl(umin(umin(f.x, f.y), f.z));
        if (ready < need) __builtin_amdgcn_s_sleep(1);
      }
    };
    wait_ready(1);
    if (wave == 0) STAMP(2);
    asm volatile("s_waitcnt vmcnt(0) lgkmcnt(0)\n\ts_nop 4" ::: "memory");
#pragma unroll
    for (int t = 0; t < 3; ++t)
#pragma unroll
      for (int j = 0; j < 4; ++j) LDB(b[t][j], t * B_IMG, j);
#pragma unroll
    for (int i = 0; i < 6; ++i) LDA(a[i], offA, i * 1024);
    asm volatile("s_waitcnt vmcnt(8) lgkmcnt(0)" ::: "memory");

    unsigned vA = offA;
    unsigned vFull = fullAddr, vFree = myFree;
    u4 fw = {0u, 0u, 0u, 0u};
    unsigned fm = 0;
    asm volatile("" : "+v"(vFull), "+v"(vFree));
    for (unsigned s4 = 0; s4 < (unsigned)NST; s4 += 4) {
      unsigned vN, soffB;
#pragma unroll
      for (int t = 0; t < 4; ++t) {
#pragma unroll
        for (int i = 0; i < 9; ++i) {
          const int buf = (t * 9 + i) % 6;
#pragma unroll
          for (int j = 0; j < 4; ++j) {
            MFMA(acc[i][j], a[buf], b[t][j]);
            SCHED();
            if (j == 3) {
              if (i < 3) LDA(a[buf], vA, t * ST + (i + 6) * 1024);
              else if (t < 3) LDA(a[buf], vA, (t + 1) * ST + (i - 3) * 1024);
              else LDA(a[buf], vN, (i - 3) * 1024);
            }
            if (i == 0 && j == 0) { soffB = (s4 + t + 3u) * B_IMG; asm volatile("" : "+s"(soffB)); }
            if (i == 0 && j == 1) asm volatile("ds_read_b128 %0, %1" : "+v"(fw) : "v"(vFull) : "memory");
            if (i == 1 && j == 0 && t == 3) {
              vN = offA + ((s4 + 4u) & (NSLOT - 1)) * ST;
              asm volatile("" : "+v"(vN));
            }
            if (i == 2 && j == 2) asm volatile("s_waitcnt lgkmcnt(2)" : "+v"(fw) :: "memory");
            if (i == 2 && j == 3) { fm = umin(umin(fw.x, fw.y), fw.z); asm volatile("" : "+v"(fm)); }
            if (i == 3 && j == 0) { ready = rfl(fm); asm volatile("" : "+s"(ready)); }
            if (i == 3 && j == 2) { if (ready < s4 + t + 2u) wait_ready(s4 + t + 2u); }
            if (i >= BPOS && i < BPOS + 4 && j == (i == 0 ? 2 : 1)) LDB(b[(t + 3) & 3][i - BPOS], soffB, i - BPOS);
            if (i == 5 && j == 2) {
              WAIT_L(2);
              if (t == 3) lds_st(vFree, s4 + 4u + NSLOT);
            }
            if (i == 8 && j == 2) asm volatile("s_waitcnt vmcnt(8) lgkmcnt(2)" ::: "memory");
            SCHED();
          }
        }
      }
      vA = vN;
    }
    asm volatile("s_waitcnt vmcnt(0) lgkmcnt(0)" ::: "memory");
    asm volatile("s_nop 7\n\ts_nop 7" ::: "memory");
    SCHED();
    const unsigned bad = rfl(lds_ld_wait(badAddr));
    float* ocol = p.out + (size_t)((lane >> 4) * 4) * N_SH + cb * BN + wave * 64 + (lane & 15) * 4;
    const float nanv = __builtin_nanf("");
#pragma unroll
    for (int i = 0; i < 9; ++i) {
      if (tr == 28 && (i < 4 || i == 8)) continue;
      float* orow = ocol + (size_t)(i < 8 ? blk * BM + i * 16 : rank * M_SH + lfrag * 16) * N_SH;
#pragma unroll
      for (int r = 0; r < 4; ++r) {
        f4 o = {acc[i][0][r], acc[i][1][r], acc[i][2][r], acc[i][3][r]};
        if (bad) o = (f4){nanv, nanv, nanv, nanv};
        asm volatile("global_store_dwordx4 %0, %1, off" EPI_BITS "\n\ts_nop 1" ::"v"(orow + (size_t)r * N_SH), "v"(o) : "memory");
      }
    }
  } else {
    __builtin_amdgcn_s_setprio(FPRIO);
    const unsigned fd = wave - 4;
    const char* gb = myws + WS_XL + (size_t)(lfrag >> 3) * NST * A_IMG;
    const char* ga = tr < 28 ? myws + WS_X16 + (size_t)(call & 1u) * X16_PAR + (size_t)blk * NST * A_IMG : gb;
    const char* fa = myws + WS_FLAGA + (size_t)(call & 1u) * FLAGA_PAR + (size_t)blk * NST * 8;
    const unsigned* fl = (const unsigned*)(myws + WS_FLAGL) + (lfrag >> 3) * NST;
    const unsigned* fw = (const unsigned*)(myws + WS_FLAGW) + cb * NST;
    unsigned* const tmo = (unsigned*)(myws + WS_TMO);
    const unsigned voff = lane * 16, voffe = EXC_OFF + lane * 2, voffl = (lfrag & 7) * 1024 + voff;
    const unsigned pl = 3u * umin(lane, 7u);
    const unsigned myFull = fullAddr + fd * 4;
    lds_st(myFull, fd);
    bool allRaw = tr == 28;
    unsigned arrA = allRaw ? 2u * NST : 0u;
    unsigned pbase = 0;
    unsigned arrL = 0;
    unsigned freed = NSLOT;
    unsigned pmeta = 0;
    u4 RA[9], RB[9];
    unsigned eA0, eA1, eB0, eB1, metaA = 0, metaB = 0, tvA = 0, tvB = 0;
#define POLL_LOAD(sf) __hip_atomic_load((const unsigned long long*)(fa + umin((sf) + pl, NST - 1) * 8u), __ATOMIC_RELAXED, __HIP_MEMORY_SCOPE_SYSTEM)
#define POLL_TAKE(pn, sf)                                                                    \
    {                                                                                        \
      const unsigned ok = (unsigned)__ballot((unsigned)(pn) >= call) & 0xffu;                \
      arrA = (sf) + 3u * (unsigned)__builtin_ctz(~ok);                                       \
      pbase = (sf);                                                                          \
      pmeta = (unsigned)((pn) >> 32);                                                        \
    }
#define FETCH(R, e0, e1, meta, tvn, sf, real)                                                     \
    {                                                                                        \
      meta = allRaw ? META_RAW : (unsigned)__builtin_amdgcn_readlane(pmeta, (((sf) - pbase) / 3u) & 63u); \
      const unsigned nrec = (real) ? (unsigned)(NST * A_IMG) : 0u;                           \
      const __amdgpu_buffer_rsrc_t ra = __builtin_amdgcn_make_buffer_rsrc((void*)ga, 0, nrec, 0x00020000); \
      const __amdgpu_buffer_rsrc_t rb = __builtin_amdgcn_make_buffer_rsrc((void*)gb, 0, nrec, 0x00020000); \
      const unsigned so = ((sf) & (NST - 1)) * A_IMG;                                        \
      tvn = __hip_atomic_load(tmo, __ATOMIC_RELAXED, __HIP_MEMORY_SCOPE_AGENT);              \
      const __amdgpu_buffer_rsrc_t rc = __builtin_amdgcn_make_buffer_rsrc((void*)ga, 0, meta == META_RAW ? nrec : 0u, 0x00020000); \
      _Pragma("unroll") for (int f = 0; f < 4; ++f) {                                        \
        R[4 + f] = __builtin_amdgcn_raw_buffer_load_b128(f < 3 ? ra : rc, voff + f * 1024, so + 4096, 17); \
        R[f] = __builtin_amdgcn_raw_buffer_load_b128(ra, voff + f * 1024, so, 17);           \
      }                                                                                      \
      e0 = __builtin_amdgcn_raw_buffer_load_b16(ra, voffe, so, 17);                          \
      e1 = __builtin_amdgcn_raw_buffer_load_b16(ra, voffe + 128, so, 17);                    \
      R[8] = __builtin_amdgcn_raw_buffer_load_b128(rb, voffl, so, 17);                       \
      CFENCE();                                                                              \
    }
    auto await = [&](unsigned sf) {
      while (arrL < umin(sf + 1u + LOOK, NST)) {
        const unsigned fi = umin(arrL + lane, NST - 1);
        arrL += poll_flags(fl + fi, fw + fi, call, myws, badAddr, lane);
      }
      if (sf >= arrA) {
        const unsigned long long t0 = __builtin_amdgcn_s_memrealtime();
        for (unsigned spin = 1;; ++spin) {
          const unsigned long long pn = POLL_LOAD(sf);
          POLL_TAKE(pn, sf)
          if (sf < arrA) break;
          if ((spin & 63u) == 0u &&
              (rfl(__hip_atomic_load(tmo, __ATOMIC_RELAXED, __HIP_MEMORY_SCOPE_AGENT)) == call ||
               __builtin_amdgcn_s_memrealtime() - t0 > DEADLINE_TICKS)) {
            if (lane == 0) __hip_atomic_store(tmo, call, __ATOMIC_RELAXED, __HIP_MEMORY_SCOPE_AGENT);
            lds_st(badAddr, 1u);
            allRaw = true;
            arrA = 2u * NST;
            break;
          }
          __builtin_amdgcn_s_sleep(POLL_SLEEP);
        }
      }
      CFENCE();
    };
#define FEED(R, e0, e1, meta, tv, Rn, en0, en1, metan, tvn)                                           \
    {                                                                                        \
      const unsigned sn = s + 3u;                                                            \
      const bool more = sn < (unsigned)NST;                                                  \
      const bool early = more && sn < arrA && arrL >= umin(sn + 1u + LOOK, NST);             \
      const unsigned sp = early ? sn + 3u : sn;          \
      FETCH(Rn, en0, en1, metan, tvn, sn, early)                                                  \
      const unsigned long long pn = POLL_LOAD(sp);                                           \
      while (freed <= s) {                             \
        const u4 f = lds_ld4_wait(freeAddr);                                                 \
        freed = rfl(umin(umin(f.x, f.y), umin(f.z, f.w)));                                   \
        if (freed <= s) __builtin_amdgcn_s_sleep(2);                                         \
      }                                                                                      \
      const unsigned slot = lds0 + (s % NSLOT) * ST;                                         \
      if (meta != META_RAW) {                                                                \
        unpack_image(R, e0, e1, slot, meta, lane);                                           \
      } else {                                                                               \
        _Pragma("unroll") for (int f = 0; f < 8; ++f)                                        \
          asm volatile("ds_write_b128 %0, %1 offset:%2" ::"v"(slot + voff), "v"(R[f]), "i"(f * 1024) : "memory"); \
      }                                                                                      \
      asm volatile("ds_write_b128 %0, %1 offset:8192" ::"v"(slot + voff), "v"(R[8]) : "memory"); \
      if (!more) {                                     \
        if (rfl(tv) == call) lds_st(badAddr, 1u);                                                 \
        WAIT_L(0);                                                                           \
        lds_st(myFull, 2u * NST);                                                            \
        break;                                                                               \
      }                                                                                      \
      WAIT_L(0);                                                                             \
      lds_st(myFull, sn);                                                                    \
      NG_STAMP()                                                                             \
      if (sp >= arrA) POLL_TAKE(pn, sp)                                                      \
      if (!early) {                                                                          \
        await(sn);                                                                           \
        FETCH(Rn, en0, en1, metan, tvn, sn, true)                                                 \
      }                                                                                      \
      s = sn;                                                                                \
    }
#define NG_STAMP()
    unsigned s = fd;
    await(s);
    FETCH(RA, eA0, eA1, metaA, tvA, s, true)
    for (;;) {
      FEED(RA, eA0, eA1, metaA, tvA, RB, eB0, eB1, metaB, tvB)
      FEED(RB, eB0, eB1, metaB, tvB, RA, eA0, eA1, metaA, tvA)
    }
    STAMP(wave + 1);
  }
}

}

extern "C" void kernel_launch(int rank, int nranks, ncclComm_t comm,
                              hipStream_t stream,
                              void* const* d_in, const int* in_sizes, int n_in,
                              void* d_out, int out_size,
                              const int* in_dtypes, int out_dtype,
                              void* const* d_out_all,
                              void* const* d_ws_all, size_t ws_bytes) {
  (void)comm; (void)in_dtypes; (void)out_dtype; (void)d_out_all;
  if (nranks != NR || (unsigned)rank >= (unsigned)NR || n_in != 2 || ws_bytes < WS_END ||
      in_sizes[0] != M_SH * KD || in_sizes[1] != KD * N_TOT || out_size != NR * M_SH * N_SH) {
    fprintf(stderr, "kernel_launch: unexpected problem shape\n");
    return;
  }
  static bool attr_set[NR] = {false, false, false, false, false, false, false, false};
  if (!attr_set[rank]) {
    HIP_CHECK(hipFuncSetAttribute((const void*)fused_kernel, hipFuncAttributeMaxDynamicSharedMemorySize, LDS_BYTES));
    attr_set[rank] = true;
  }
  Params p;
  p.x = (const float*)d_in[0];
  p.w = (const float*)d_in[1];
  p.out = (float*)d_out;
  for (int i = 0; i < NR; ++i) p.ws[i] = (char*)d_ws_all[i];
  p.rank = rank;
  p.pad = 0;
  hipLaunchKernelGGL(fused_kernel, dim3(NBLK), dim3(512), LDS_BYTES, stream, p);
  HIP_CHECK(hipGetLastError());
}
```

```cpp
#include <hip/hip_runtime.h>
#include <rccl/rccl.h>
#include <cstdio>

#define HIP_CHECK(expr) do { hipError_t _e = (expr); if (_e != hipSuccess) { \
    fprintf(stderr, "HIP error %s at %s:%d\n", hipGetErrorString(_e), __FILE__, __LINE__); \
    return; } } while (0)

#define LAS __attribute__((address_space(3)))
#define GAS __attribute__((address_space(1)))
typedef _Float16 h8 __attribute__((ext_vector_type(8)));
typedef _Float16 h2 __attribute__((ext_vector_type(2)));
typedef float f4 __attribute__((ext_vector_type(4)));
typedef float f2 __attribute__((ext_vector_type(2)));
typedef unsigned u4 __attribute__((ext_vector_type(4)));
typedef unsigned u2 __attribute__((ext_vector_type(2)));
typedef unsigned short us2 __attribute__((ext_vector_type(2)));

namespace {

constexpr int NR = 8;
constexpr int M_SH = 512, KD = 4096, N_SH = 2048, N_TOT = 16384;
constexpr int BM = 128, BN = 256, BK = 32;
constexpr int NST = KD / BK;
constexpr int A_IMG = BM * BK * 2;
constexpr int B_IMG = BN * BK * 2;
constexpr int ST = A_IMG + 1024;
constexpr int NSLOT = 16;
constexpr int CTRL = NSLOT * ST;
constexpr int LDS_BYTES = CTRL + 1024;
constexpr int NBLK = 256;
#define PPX 3
#define NPACT (8 * PPX)
constexpr int NPWG = NPACT;
constexpr int NGWG = 232;
constexpr int NITEM = 1280;
constexpr int NUNIT = 4 * NST;
constexpr int PRING = 16;
#define PWIN 2
#define WFRAC 0x70
#define NWIDE 0
#define PLEAD 0u
#define BPOS 0
#define LEAD0 8
#define LEAD1 0
#define POLL_SLEEP 1
#define RAWTAIL 2
#define CPRIO 2
#define FPRIO 1
#define EPI_BITS " sc0 sc1"
#define XP_AUX 0
#define WBATCH 2
#define CONV_TICKS 3
constexpr int LOOK = 3;
#define S4 96

constexpr size_t WS_CALL = 0;
constexpr size_t WS_TMO = 32768;
constexpr size_t WS_TICKET = 32768 + 256;
constexpr size_t WS_FLAGA = 65536;
constexpr size_t FLAGA_PAR = (size_t)NR * 4 * NST * 8;
constexpr size_t WS_FLAGW = WS_FLAGA + 2 * FLAGA_PAR;
constexpr size_t WS_FLAGL = WS_FLAGW + 8 * NST * 4;
constexpr size_t WS_DBG = 262144;
constexpr size_t WS_X16 = 1u << 20;
constexpr size_t X16_PAR = (size_t)NR * 4 * NST * A_IMG;
constexpr size_t WS_W16 = WS_X16 + 2 * X16_PAR;
constexpr size_t WS_XL = WS_W16 + (size_t)8 * NST * B_IMG;
constexpr size_t WS_END = WS_XL + (size_t)4 * NST * A_IMG;

constexpr unsigned META_RAW = 0x80000000u;
constexpr int EXC_OFF = 7168;
constexpr unsigned MAXEXC = 128;

constexpr unsigned long long DEADLINE_TICKS = 40000000ull;

struct Params {
  const float* x;
  const float* w;
  float* out;
  char* ws[NR];
  int rank;
  int pad;
};

#define SCHED() __builtin_amdgcn_sched_barrier(0)
#define CFENCE() asm volatile("" ::: "memory")
#define WAIT_V(n) asm volatile("s_waitcnt vmcnt(%0)" ::"n"(n) : "memory")
#define WAIT_L(n) asm volatile("s_waitcnt lgkmcnt(%0)" ::"n"(n) : "memory")

__device__ __forceinline__ unsigned rfl(unsigned v) { return __builtin_amdgcn_readfirstlane(v); }
__device__ __forceinline__ unsigned umin(unsigned a, unsigned b) { return a < b ? a : b; }
__device__ __forceinline__ unsigned umax(unsigned a, unsigned b) { return a > b ? a : b; }

__device__ __forceinline__ unsigned cvt2(float a, float b) {
  f2 v = {a, b};
  h2 h = __builtin_convertvector(v, h2);
  return __builtin_bit_cast(unsigned, h);
}

__device__ __forceinline__ unsigned image_of(unsigned o) {
  constexpr unsigned MID = NST - LEAD0 - LEAD1;
  if (o < LEAD0) return o;
  o -= LEAD0;
  if (o < 2u * LEAD1) return (o & 1u) ? NST + (o >> 1) : LEAD0 + (o >> 1);
  o -= 2u * LEAD1;
  if (o < 4u * MID) {
    const unsigned rb = o & 3u;
    return rb * NST + (o >> 2) + (rb == 0u ? LEAD0 + LEAD1 : rb == 1u ? LEAD1 : 0u);
  }
  o -= 4u * MID;
  if (o < 3u * LEAD0) {
    const unsigned rb = 1u + o % 3u;
    return rb * NST + MID + o / 3u + (rb == 1u ? LEAD1 : 0u);
  }
  o -= 3u * LEAD0;
  return (2u + (o & 1u)) * NST + NST - LEAD1 + (o >> 1);
}

__device__ __forceinline__ void lds_st(unsigned addr, unsigned v) {
  asm volatile("ds_write_b32 %0, %1" ::"v"(addr), "v"(v) : "memory");
}
__device__ __forceinline__ u4 lds_ld4_wait(unsigned addr) {
  u4 r;
  asm volatile("ds_read_b128 %0, %1\n\ts_waitcnt lgkmcnt(0)" : "=&v"(r) : "v"(addr) : "memory");
  return r;
}
__device__ __forceinline__ unsigned lds_ld_wait(unsigned addr) {
  unsigned r;
  asm volatile("ds_read_b32 %0, %1\n\ts_waitcnt lgkmcnt(0)" : "=&v"(r) : "v"(addr) : "memory");
  return r;
}
__device__ __forceinline__ void flag_store(const char* sbase, unsigned voff, unsigned val) {
  unsigned long long keep;
  asm volatile(
      "s_mov_b64 %0, exec\n\t"
      "s_mov_b64 exec, 1\n\t"
      "s_nop 2\n\t"
      "global_store_dword %1, %2, %3 sc0 sc1\n\t"
      "s_mov_b64 exec, %0"
      : "=&s"(keep)
      : "v"(voff), "v"(val), "s"(sbase)
      : "memory");
}
__device__ __forceinline__ void flag_store2(const char* sbase, unsigned voff, unsigned lo, unsigned hi) {
  unsigned long long keep;
  const u2 v = {lo, hi};
  asm volatile(
      "s_mov_b64 %0, exec\n\t"
      "s_mov_b64 exec, 1\n\t"
      "s_nop 2\n\t"
      "global_store_dwordx2 %1, %2, %3 sc0 sc1\n\t"
      "s_mov_b64 exec, %0"
      : "=&s"(keep)
      : "v"(voff), "v"(v), "s"(sbase)
      : "memory");
}
__device__ __forceinline__ unsigned perm(unsigned hi, unsigned lo, unsigned sel) { return __builtin_amdgcn_perm(hi, lo, sel); }
__device__ __forceinline__ unsigned xad(unsigned a, unsigned b, unsigned c) {
  unsigned r;
  asm("v_xad_u32 %0, %1, %2, %3" : "=v"(r) : "v"(a), "v"(b), "s"(c));
  return r;
}

__device__ __forceinline__ unsigned pack_image(const unsigned (&D)[32], unsigned slot, int lane, bool force_raw,
                                               unsigned& base) {
  unsigned meta = META_RAW;
  for (int attempt = 0; attempt < 2 && !force_raw; ++attempt) {
    const unsigned bb = base * 0x04040404u;
    unsigned T[16], Q[4], c = 0;
#pragma unroll
    for (int g = 0; g < 4; ++g) {
      unsigned dl[4];
#pragma unroll
      for (int k = 0; k < 4; ++k) {
        const unsigned H = perm(D[8 * g + 2 * k + 1], D[8 * g + 2 * k], 0x07050301u);
        const unsigned R = (H | 0x80808080u) - bb;
        T[4 * g + k] = (R & 0x1f1f1f1fu) | (H & 0x80808080u);
        dl[k] = H ^ (T[4 * g + k] + bb);
      }
      Q[g] = (dl[0] >> 5) | (dl[1] >> 3) | (dl[2] >> 1) | (dl[3] << 1);
      c += (unsigned)__builtin_popcount((Q[g] | (Q[g] >> 1)) & 0x55555555u);
    }
    unsigned cnt = 0, ix = 0;
#pragma unroll
    for (int bt = 0; bt < 3; ++bt) {
      const unsigned long long m = __ballot((c >> bt) & 1u);
      ix += __builtin_amdgcn_mbcnt_hi((unsigned)(m >> 32), __builtin_amdgcn_mbcnt_lo((unsigned)m, 0u)) << bt;
      cnt += (unsigned)__builtin_popcountll(m) << bt;
    }
    if (__ballot(c > 7u) != 0ull) cnt = MAXEXC + 1;
    if (cnt > 96u && attempt == 0) {
      us2 mx = {0, 0};
      unsigned am = 0x7fff7fffu;
      asm volatile("" : "+s"(am));
#pragma unroll
      for (int i = 0; i < 32; ++i) mx = __builtin_elementwise_max(mx, __builtin_bit_cast(us2, D[i] & am));
      const unsigned el = (unsigned)(mx.x > mx.y ? mx.x : mx.y) >> 10;
      unsigned lo = 7, hi = 31;
      while (lo < hi) {
        const unsigned mid = (lo + hi) >> 1;
        if (__builtin_popcountll(__ballot(el > mid)) <= 20) hi = mid; else lo = mid + 1;
      }
      base = lo - 7;
      continue;
    }
    if (cnt > MAXEXC) break;
    meta = base | (cnt << 8);
    unsigned ea = slot + EXC_OFF + ix * 2u;
#pragma unroll
    for (int g = 0; g < 4; ++g) {
      unsigned q = Q[g];
      while (q != 0u) {
        const unsigned pos = (unsigned)__builtin_ctz(q) & ~1u;
        const unsigned dv = (q >> pos) & 3u;
        q &= ~(3u << pos);
        const unsigned hw = (4u * g + ((pos >> 1) & 3u)) * 256u + (unsigned)lane * 4u + (pos >> 3);
        *(LAS unsigned short*)(size_t)ea = (unsigned short)(hw | (dv << 12));
        ea += 2u;
      }
    }
    const unsigned la = slot + lane * 16;
    unsigned P[12];
#pragma unroll
    for (int g = 0; g < 4; ++g) {
      u4 Lp;
#pragma unroll
      for (int k = 0; k < 4; ++k) Lp[k] = perm(D[8 * g + 2 * k + 1], D[8 * g + 2 * k], 0x06040200u);
      *(LAS u4*)(size_t)(la + g * 1024) = Lp;
      const unsigned t3 = T[4 * g + 3];
      P[3 * g + 0] = T[4 * g + 0] | ((t3 << 5) & 0x60606060u);
      P[3 * g + 1] = T[4 * g + 1] | ((t3 << 3) & 0x60606060u);
      P[3 * g + 2] = T[4 * g + 2] | ((t3 << 1) & 0x20202020u) | ((t3 >> 1) & 0x40404040u);
    }
#pragma unroll
    for (int t = 0; t < 3; ++t)
      *(LAS u4*)(size_t)(la + (4 + t) * 1024) = (u4){P[4 * t], P[4 * t + 1], P[4 * t + 2], P[4 * t + 3]};
    break;
  }
  if (meta == META_RAW) {
#pragma unroll
    for (int j = 0; j < 16; ++j) *(LAS u2*)(size_t)(slot + j * 512 + lane * 8) = (u2){D[2 * j], D[2 * j + 1]};
  }
  return meta;
}

__device__ __forceinline__ void unpack_image(const u4 (&R)[9], unsigned e0, unsigned e1, unsigned slot, unsigned meta,
                                             int lane) {
  const unsigned la = slot + lane * 8;
  const unsigned bb = (meta & 31u) * 0x04040404u;
  const unsigned cnt = meta >> 8;
  unsigned H[16], X[12], A[4], S[4];
#pragma unroll
  for (int i = 0; i < 12; ++i) X[i] = R[4 + (i >> 2)][i & 3] & 0x60606060u;
  SCHED();
#pragma unroll
  for (int i = 0; i < 12; ++i) H[(i / 3) * 4 + i % 3] = xad(R[4 + (i >> 2)][i & 3], X[i], bb);
  SCHED();
#pragma unroll
  for (int i = 0; i < 12; ++i) { X[i] >>= 5 - 2 * (i % 3); asm volatile("" : "+v"(X[i])); }
  SCHED();
  const unsigned e = (unsigned)lane < cnt ? e0 : 0u;
#pragma unroll
  for (int g = 0; g < 4; ++g) { A[g] = X[3 * g] | X[3 * g + 1] | X[3 * g + 2]; S[g] = X[3 * g + 2] & 0x20202020u; }
  SCHED();
#pragma unroll
  for (int g = 0; g < 4; ++g) A[g] = xad(A[g], S[g], bb);
  const unsigned xa = slot + ((e & 0xffeu) << 1), xm = (e >> 12) << (13u + ((e & 1u) << 4));
  SCHED();
#pragma unroll
  for (int g = 0; g < 4; ++g) H[4 * g + 3] = A[g] | (S[g] << 2);
  SCHED();
  u2 o[17];
#pragma unroll
  for (int j = 0; j < 17; ++j) {
    if (j < 16) {
      o[j] = (u2){perm(H[j], R[j >> 2][j & 3], 0x05010400u), perm(H[j], R[j >> 2][j & 3], 0x07030602u)};
      asm volatile("" : "+v"(o[j]));
    }
    if (j >= 1)
      asm volatile("ds_write_b64 %0, %1 offset:%2" ::"v"(la), "v"(o[j - 1]), "i"((j - 1) * 512), "v"(o[j >= 2 ? j - 2 : 0]) : "memory");
  }
  asm volatile("ds_xor_b32 %0, %1" ::"v"(xa), "v"(xm) : "memory");
  if (cnt > 64u) {
    const unsigned f = (unsigned)lane + 64u < cnt ? e1 : 0u;
    asm volatile("ds_xor_b32 %0, %1" ::"v"(slot + ((f & 0xffeu) << 1)), "v"((f >> 12) << (13u + ((f & 1u) << 4))) : "memory");
  }
}

__device__ __forceinline__ void touch(const void* addr, unsigned lds_scratch) {
  unsigned keep;
  asm volatile("s_mov_b32 %0, m0\n\ts_mov_b32 m0, %2\n\ts_nop 0\n\tglobal_load_lds_dword %1, off sc0 sc1\n\ts_mov_b32 m0, %0"
               : "=&s"(keep) : "v"(addr), "s"(lds_scratch) : "memory");
}

#define STORE16_WT(voff, data, sbase, imm)                                                   \
  asm volatile("global_store_dwordx4 %0, %1, %2 offset:%3 sc0 sc1\n\ts_nop 1"                \
               ::"v"(voff), "v"(data), "s"(sbase), "i"(imm) : "memory")

#define STAMP(i) do {} while (0)

__device__ __forceinline__ unsigned poll_flags(const unsigned* f0, const unsigned* f1, unsigned call, char* myws,
                                               unsigned badAddr, int lane) {
  unsigned* const tmo = (unsigned*)(myws + WS_TMO);
  const unsigned long long t0 = __builtin_amdgcn_s_memrealtime();
  for (unsigned spin = 1;; ++spin) {
    const unsigned f = umin(__hip_atomic_load(f0, __ATOMIC_RELAXED, __HIP_MEMORY_SCOPE_SYSTEM),
                            __hip_atomic_load(f1, __ATOMIC_RELAXED, __HIP_MEMORY_SCOPE_SYSTEM));
    const unsigned long long ok = __ballot(f >= call);
    const unsigned n = ok == ~0ull ? 64u : (unsigned)__builtin_ctzll(~ok);
    if (n) return n;
    if ((spin & 63u) == 0u &&
        (rfl(__hip_atomic_load(tmo, __ATOMIC_RELAXED, __HIP_MEMORY_SCOPE_AGENT)) == call ||
         __builtin_amdgcn_s_memrealtime() - t0 > DEADLINE_TICKS)) {
      if (lane == 0) __hip_atomic_store(tmo, call, __ATOMIC_RELAXED, __HIP_MEMORY_SCOPE_AGENT);
      lds_st(badAddr, 1u);
      return 64u;
    }
    __builtin_amdgcn_s_sleep(2);
  }
}

__global__ __launch_bounds__(512, 2) void fused_kernel(Params p) {
  extern __shared__ __attribute__((aligned(1024))) char smem[];
  const unsigned lds0 = (unsigned)(unsigned long long)(LAS char*)smem;
  const int tid = threadIdx.x;
  const int lane = tid & 63;
  const int wave = (int)rfl((unsigned)(tid >> 6));
  const int bid = blockIdx.x;
  const int xcd = bid & 7, idx = bid >> 3;
  int rank = p.rank;
  unsigned long long a0 = (unsigned long long)p.ws[0], a1 = (unsigned long long)p.ws[wave & (NR - 1)],
                     a2 = (unsigned long long)p.x;
  asm volatile("" : "+s"(rank), "+s"(a0), "+s"(a1), "+s"(a2));
  char* const myws = (char*)(GAS char*)a0;
  char* const dws = (char*)(GAS char*)a1;
  const float* const px = (const float*)(GAS const float*)a2;

  unsigned* const callp = (unsigned*)(myws + WS_CALL + (size_t)bid * 128) + wave;

  const unsigned prow = lane >> 2;
  const unsigned pko = (lane & 3) ^ ((prow >> 3) << 1);

#define XCVT(L, f) ((u4){cvt2(L[2 * (f)][0], L[2 * (f)][1]), cvt2(L[2 * (f)][2], L[2 * (f)][3]), \
                         cvt2(L[2 * (f) + 1][0], L[2 * (f) + 1][1]), cvt2(L[2 * (f) + 1][2], L[2 * (f) + 1][3])})
  constexpr bool kRaw = false;

  const __amdgpu_buffer_rsrc_t xr = __builtin_amdgcn_make_buffer_rsrc((void*)px, 0, M_SH * KD * 4, 0x00020000);
  const unsigned xo0 = (lane >> 3) * (KD * 4) + (lane & 7) * 16, xo1 = xo0 ^ 64u;
#define PLOAD(L, o)                                                                          \
      {                                                                                      \
        const unsigned im_ = image_of(o);                                                    \
        const unsigned sb = im_ / NST * (BM * KD * 4) + im_ % NST * (BK * 4);                \
        _Pragma("unroll") for (int j = 0; j < 16; ++j)                                       \
          L[j] = __builtin_bit_cast(f4, __builtin_amdgcn_raw_buffer_load_b128(xr, (j & 1) ? xo1 : xo0, sb + j * (8 * KD * 4), XP_AUX)); \
      }
  const unsigned callv = __hip_atomic_load(callp, __ATOMIC_RELAXED, __HIP_MEMORY_SCOPE_AGENT);
  if (tid < 64) ((LAS unsigned*)(smem + CTRL))[tid] = 0u;
#define ENTRY_BARRIER() do { WAIT_L(0); __builtin_amdgcn_s_barrier(); CFENCE(); } while (0)
#define STAMP0() do {} while (0)

  if (idx < PPX) {
    if (bid >= NPWG) return;
    f4 L0[2];
    {
      const unsigned im = image_of((unsigned)bid);
      const float* sb = px + (size_t)(im / NST * BM + wave * 16 + prow) * KD + im % NST * BK + pko * 8;
      L0[0] = *(const f4*)sb;
      L0[1] = *(const f4*)(sb + 4);
    }
    const unsigned convAddr = lds0 + CTRL;
    const unsigned sentAddr = lds0 + CTRL + 32;
    const unsigned uidAddr = lds0 + CTRL + 64;
    const unsigned voff = lane * 16;
    if (wave == 0) {
#define PPUB(u, im, meta)                                                                    \
      {                                                                                      \
        lds_st(uidAddr + ((u) % PRING) * 8, (im));                                           \
        lds_st(uidAddr + ((u) % PRING) * 8 + 4, (meta));                                     \
        WAIT_L(0);                                                                           \
        lds_st(convAddr, (u) + 1u);                                                          \
      }
#define PCVT(D, L)                                                                           \
      _Pragma("unroll") for (int j = 0; j < 16; ++j) {                                       \
        D[2 * j] = cvt2(L[j][0], L[j][1]);                                                   \
        D[2 * j + 1] = cvt2(L[j][2], L[j][3]);                                               \
        asm volatile("" : "+v"(D[2 * j]), "+v"(D[2 * j + 1]));                               \
      }
#define PTICKET(tk)                                                                          \
      {                                                                                      \
        unsigned long long keep;                                                             \
        asm volatile("s_mov_b64 %1, exec\n\ts_mov_b64 exec, 1\n\ts_nop 2\n\t"                \
                     "global_atomic_add %0, %2, %3, %4 sc0\n\ts_mov_b64 exec, %1"            \
                     : "=&v"(tk), "=&s"(keep) : "v"(0u), "v"(1u), "s"(ticket) : "memory");   \
      }
      f4 Lx[16];
      unsigned D[32], base = 24;
      PLOAD(Lx, (unsigned)(bid + NPWG))
      SCHED();
      *(LAS u4*)(size_t)(lds0 + voff) = XCVT(L0, 0);
      ENTRY_BARRIER();
      STAMP0();
      const unsigned call = rfl(callv) + 1u;
      unsigned* const ticket = (unsigned*)(myws + WS_TICKET);
      const unsigned tbase = (call - 1u) * 1024u;
      if (lane == 0) {
        __hip_atomic_store(callp, call, __ATOMIC_RELAXED, __HIP_MEMORY_SCOPE_AGENT);
        __hip_atomic_fetch_max(ticket, tbase, __ATOMIC_RELAXED, __HIP_MEMORY_SCOPE_AGENT);
      }
      __builtin_amdgcn_s_setprio(1);
      unsigned o = bid + NPWG, tk = tbase + bid - NPWG;
      for (unsigned u = 1;; ++u) {
        for (;;) {
          const u4 c0 = lds_ld4_wait(sentAddr), c1 = lds_ld4_wait(sentAddr + 16);
          const unsigned mn = rfl(umin(umin(umin(c0.y, c0.z), umin(c0.w, c1.x)), umin(umin(c1.y, c1.z), c1.w)));
          const unsigned mx = rfl(umax(umax(umax(c0.y, c0.z), umax(c0.w, c1.x)), umax(umax(c1.y, c1.z), c1.w)));
          if (mx + PLEAD >= u && mn + PRING > u + 1u) break;
          __builtin_amdgcn_s_sleep(2);
        }
        asm volatile("s_waitcnt vmcnt(0)" : "+v"(tk) :: "memory");
        __builtin_amdgcn_s_waitcnt(0x0f70);
        SCHED();
        PCVT(D, Lx)
        const unsigned on = rfl(tk) - tbase + 3u * NPWG;
        PLOAD(Lx, umin(on, NUNIT - 1))
        if (on < (unsigned)NUNIT) PTICKET(tk)
        SCHED();
        const unsigned meta = pack_image(D, lds0 + (u % PRING) * A_IMG, lane,
                                         kRaw || image_of(o) % NST >= (unsigned)(NST - RAWTAIL), base);
        PPUB(u, image_of(o), meta)
        if (on >= (unsigned)NUNIT) {
          PPUB(u + 1u, 0xffffffffu, 0u)
          break;
        }
        o = on;
      }
      STAMP(16);
    } else {
      *(LAS u4*)(size_t)(lds0 + wave * 1024 + voff) = XCVT(L0, 0);
      ENTRY_BARRIER();
      const unsigned call = rfl(callv) + 1u;
      const char* xd = dws + WS_X16 + (size_t)(call & 1u) * X16_PAR + (size_t)(rank * 4) * NST * A_IMG;
      const char* fd = dws + WS_FLAGA + (size_t)(call & 1u) * FLAGA_PAR + (size_t)(rank * 4) * NST * 8;
      const unsigned voff2 = voff + 4096;
      unsigned conv = 1, prev = 0, prevMeta = 0;
      for (unsigned u = 0;; ++u) {
        while (conv <= u) {
          conv = rfl(lds_ld_wait(convAddr));
          if (conv <= u) __builtin_amdgcn_s_sleep(1);
        }
        const unsigned la = lds0 + (u % PRING) * A_IMG + voff;
        u4 d[8];
        u2 id;
        asm volatile(
            "ds_read_b64 %8, %10\n\tds_read_b128 %7, %9 offset:7168\n\t"
            "ds_read_b128 %0, %9\n\tds_read_b128 %1, %9 offset:1024\n\tds_read_b128 %2, %9 offset:2048\n\t"
            "ds_read_b128 %3, %9 offset:3072\n\tds_read_b128 %4, %9 offset:4096\n\tds_read_b128 %5, %9 offset:5120\n\t"
            "ds_read_b128 %6, %9 offset:6144\n\ts_waitcnt lgkmcnt(0)"
            : "=&v"(d[0]), "=&v"(d[1]), "=&v"(d[2]), "=&v"(d[3]), "=&v"(d[4]), "=&v"(d[5]), "=&v"(d[6]), "=&v"(d[7]), "=&v"(id)
            : "v"(la), "v"(uidAddr + (u % PRING) * 8) : "memory");
        lds_st(sentAddr + wave * 4, u + 1u);
        const unsigned im = u ? rfl(id.x) : image_of((unsigned)bid);
        if (im >= (unsigned)NUNIT) break;
        const unsigned meta = u ? rfl(id.y) : META_RAW;
        const unsigned nl = meta == META_RAW ? 64u : umax((((meta >> 8) + 31u) >> 5) << 2, 4u);
        const unsigned long long lastMask = nl == 64u ? ~0ull : (1ull << nl) - 1ull;
        const char* db = xd + (size_t)im * A_IMG;
        {
          unsigned long long keep;
          asm volatile("s_mov_b64 %0, exec\n\ts_mov_b64 exec, %4\n\ts_nop 2\n\t"
                       "global_store_dwordx4 %1, %2, %3 offset:3072 sc0 sc1\n\ts_nop 1\n\ts_mov_b64 exec, %0"
                       : "=&s"(keep) : "v"(voff2), "v"(d[7]), "s"(db), "s"(lastMask) : "memory");
        }
#define SEND(W)                                                                              \
        _Pragma("unroll") for (int f = 0; f < 7; ++f) {                                      \
          WAIT_V((W) - 1 + (f < 2 * (W) - 1 ? 1 : 0) + (f != (W) - 1 ? (WFRAC >> f) & 1 : 0)); \
          if (f == (W) - 1 && u > 0)                   \
            flag_store2(fd + (size_t)prev * 8, 0u, call, prevMeta);                          \
          STORE16_WT(f < 4 ? voff : voff2, d[f], db, (f & 3) * 1024);                        \
        }
        if (bid < NWIDE) { SEND(PWIN + 1) } else { SEND(PWIN) }
        prev = im;
        prevMeta = meta;
      }
      WAIT_V(0);
      flag_store2(fd + (size_t)prev * 8, 0u, call, prevMeta);
      if (lane == 0) __hip_atomic_store(callp, call, __ATOMIC_RELAXED, __HIP_MEMORY_SCOPE_AGENT);
      STAMP(16 + wave);
    }
    return;
  }

  const int gl = (xcd >> 1) * 29 + (idx - PPX);
  const int tr = gl >> 2;
  const int cb = (xcd & 1) * 4 + (gl & 3);
  const int gidx = (idx - PPX) * 8 + xcd;
  const int blk = tr < 28 ? (((rank + 1 + (tr >> 2)) & 7) * 4 + (tr & 3)) : rank * 4 + 3;
  const int lfrag = tr < 28 ? tr : 24;
  const unsigned fullAddr = lds0 + CTRL;
  const unsigned freeAddr = lds0 + CTRL + 16;
  const unsigned badAddr = lds0 + CTRL + 32;

  const unsigned long long tstart = __builtin_amdgcn_s_memrealtime();
  ENTRY_BARRIER();
  STAMP0();
  const unsigned call = rfl(callv) + 1u;
  if (lane == 0) __hip_atomic_store(callp, call, __ATOMIC_RELAXED, __HIP_MEMORY_SCOPE_AGENT);

  if (wave >= 4) lds_st(fullAddr + (wave - 4) * 4, wave < 7 ? wave - 4 : S4 + 3);
  if (wave == 7) {
    const unsigned voff = lane * 16;
    for (int item = gidx; item < NITEM; item += NGWG) {
      while (__builtin_amdgcn_s_memrealtime() - tstart < (unsigned long long)(item * CONV_TICKS)) __builtin_amdgcn_s_sleep(8);
      const int pair = item / 20, r = item % 20;
      if (r < 16) {
        const int wcb = r & 7, wks = pair * 2 + (r >> 3);
        const float* wsrc = p.w + (size_t)(wks * BK + (lane >> 4) * 8) * N_TOT + (size_t)rank * N_SH + wcb * BN + (lane & 15) * 4;
        const char* dst = myws + WS_W16 + ((size_t)wcb * NST + wks) * B_IMG;
#pragma unroll
        for (int g0 = 0; g0 < 4; g0 += WBATCH) {
          f4 v[WBATCH][8];
#pragma unroll
          for (int g = 0; g < WBATCH; ++g)
#pragma unroll
            for (int t = 0; t < 8; ++t) v[g][t] = __builtin_nontemporal_load((const f4*)(wsrc + (size_t)t * N_TOT + (g0 + g) * 64));
          SCHED();
#pragma unroll
          for (int g = 0; g < WBATCH; ++g) {
            const char* dg = dst + (g0 + g) * 4096;
#pragma unroll
            for (int j = 0; j < 4; ++j) {
              u4 o;
              o.x = cvt2(v[g][0][j], v[g][1][j]);
              o.y = cvt2(v[g][2][j], v[g][3][j]);
              o.z = cvt2(v[g][4][j], v[g][5][j]);
              o.w = cvt2(v[g][6][j], v[g][7][j]);
              STORE16_WT(voff, o, dg, j * 1024);
            }
          }
        }
        WAIT_V(0);
        flag_store(myws + WS_FLAGW + (size_t)(wcb * NST + wks) * 4, 0u, call);
      } else {
        const unsigned im = (r - 16) * NST + pair * 2;
        const float* sb = px + (size_t)((r - 16) * BM + prow) * KD + pair * 2 * BK + pko * 8;
        const char* dst = myws + WS_XL + (size_t)im * A_IMG;
#pragma unroll
        for (int f0 = 0; f0 < 8; f0 += 2 * WBATCH) {
          f4 L[2][4 * WBATCH];
#pragma unroll
          for (int f = 0; f < 2 * WBATCH; ++f)
#pragma unroll
            for (int h = 0; h < 2; ++h) {
              L[h][2 * f] = *(const f4*)(sb + (size_t)(f0 + f) * 16 * KD + h * BK);
              L[h][2 * f + 1] = *(const f4*)(sb + (size_t)(f0 + f) * 16 * KD + h * BK + 4);
            }
          SCHED();
#pragma unroll
          for (int h = 0; h < 2; ++h)
#pragma unroll
            for (int f = 0; f < 2 * WBATCH; ++f)
              STORE16_WT(voff, XCVT(L[h], f), dst + h * A_IMG + ((f0 + f) >> 2) * 4096, ((f0 + f) & 3) * 1024);
        }
        WAIT_V(0);
        flag_store2(myws + WS_FLAGL + (size_t)im * 4, 0u, call, call);
      }
    }
    STAMP(1);
  }

  if (wave < 4) {
    __builtin_amdgcn_s_setprio(CPRIO);
    const unsigned arow = lane & 15, ako = lane >> 4;
    const unsigned offA = lds0 + arow * 64 + ((ako ^ ((arow >> 3) << 1)) << 4);
    const unsigned myFree = freeAddr + wave * 4;
    u4 wrs;
    {
      const unsigned long long wb = (unsigned long long)(myws + WS_W16 + (size_t)cb * NST * B_IMG);
      wrs.x = rfl((unsigned)wb);
      wrs.y = rfl((unsigned)(wb >> 32)) & 0xffffu;
      wrs.z = NST * B_IMG;
      wrs.w = 0x00020000u;
    }
    const unsigned voffB = wave * 4096 + lane * 16;

    f4 acc[9][4];
#pragma unroll
    for (int i = 0; i < 9; ++i)
#pragma unroll
      for (int j = 0; j < 4; ++j) {
        acc[i][j] = (f4){0.f, 0.f, 0.f, 0.f};
        asm volatile("" : "+v"(acc[i][j]));
      }
    u4 a[6], b[4][4];
#pragma unroll
    for (int i = 0; i < 6; ++i) a[i] = (u4){0u, 0u, 0u, 0u};
#pragma unroll
    for (int i = 0; i < 16; ++i) b[i >> 2][i & 3] = (u4){0u, 0u, 0u, 0u};

#define LDA(dst, addr, imm) asm volatile("ds_read_b128 %0, %1 offset:%2" : "+v"(dst) : "v"(addr), "i"(imm) : "memory")
#define LDB(dst, soff, j) asm volatile("buffer_load_dwordx4 %0, %1, %2, %3 offen offset:%4 sc0 sc1" \
                                       : "+v"(dst) : "v"(voffB), "s"(wrs), "s"(soff), "i"((j) * 1024) : "memory")
#define MFMA(c, av, bv) asm volatile("v_mfma_f32_16x16x32_f16 %0, %1, %2, %0" : "+v"(c) : "v"(av), "v"(bv))
    unsigned ready = 0;
    auto wait_ready = [&](unsigned need) {
      while (ready < need) {
        const u4 f = lds_ld4_wait(fullAddr);
        ready = rfl(umin(umin(f.x, f.y), umin(f.z, f.w)));
        if (ready < need) __builtin_amdgcn_s_sleep(1);
      }
    };
    wait_ready(1);
    if (wave == 0) STAMP(2);
    asm volatile("s_waitcnt vmcnt(0) lgkmcnt(0)\n\ts_nop 4" ::: "memory");
#pragma unroll
    for (int t = 0; t < 3; ++t)
#pragma unroll
      for (int j = 0; j < 4; ++j) LDB(b[t][j], t * B_IMG, j);
#pragma unroll
    for (int i = 0; i < 6; ++i) LDA(a[i], offA, i * 1024);
    asm volatile("s_waitcnt vmcnt(8) lgkmcnt(0)" ::: "memory");

    unsigned vA = offA;
    unsigned vFull = fullAddr, vFree = myFree;
    u4 fw = {0u, 0u, 0u, 0u};
    unsigned fm = 0;
    asm volatile("" : "+v"(vFull), "+v"(vFree));
    for (unsigned s4 = 0; s4 < (unsigned)NST; s4 += 4) {
      unsigned vN, soffB;
#pragma unroll
      for (int t = 0; t < 4; ++t) {
#pragma unroll
        for (int i = 0; i < 9; ++i) {
          const int buf = (t * 9 + i) % 6;
#pragma unroll
          for (int j = 0; j < 4; ++j) {
            MFMA(acc[i][j], a[buf], b[t][j]);
            SCHED();
            if (j == 3) {
              if (i < 3) LDA(a[buf], vA, t * ST + (i + 6) * 1024);
              else if (t < 3) LDA(a[buf], vA, (t + 1) * ST + (i - 3) * 1024);
              else LDA(a[buf], vN, (i - 3) * 1024);
            }
            if (i == 0 && j == 0) { soffB = (s4 + t + 3u) * B_IMG; asm volatile("" : "+s"(soffB)); }
            if (i == 0 && j == 1) asm volatile("ds_read_b128 %0, %1" : "+v"(fw) : "v"(vFull) : "memory");
            if (i == 1 && j == 0 && t == 3) {
              vN = offA + ((s4 + 4u) & (NSLOT - 1)) * ST;
              asm volatile("" : "+v"(vN));
            }
            if (i == 2 && j == 2) asm volatile("s_waitcnt lgkmcnt(2)" : "+v"(fw) :: "memory");
            if (i == 2 && j == 2) { fm = umin(umin(fw.x, fw.y), fw.z); asm volatile("" : "+v"(fm)); }
            if (i == 2 && j == 3) { fm = umin(fm, fw.w); asm volatile("" : "+v"(fm)); }
            if (i == 3 && j == 0) { ready = rfl(fm); asm volatile("" : "+s"(ready)); }
            if (i == 3 && j == 2) { if (ready < s4 + t + 2u) wait_ready(s4 + t + 2u); }
            if (i >= BPOS && i < BPOS + 4 && j == (i == 0 ? 2 : 1)) LDB(b[(t + 3) & 3][i - BPOS], soffB, i - BPOS);
            if (i == 5 && j == 2) {
              WAIT_L(2);
              if (t == 3) lds_st(vFree, s4 + 4u + NSLOT);
            }
            if (i == 8 && j == 2) asm volatile("s_waitcnt vmcnt(8) lgkmcnt(2)" ::: "memory");
            SCHED();
          }
        }
      }
      vA = vN;
    }
    asm volatile("s_waitcnt vmcnt(0) lgkmcnt(0)" ::: "memory");
    asm volatile("s_nop 7\n\ts_nop 7" ::: "memory");
    SCHED();
    const unsigned bad = rfl(lds_ld_wait(badAddr));
    float* ocol = p.out + (size_t)((lane >> 4) * 4) * N_SH + cb * BN + wave * 64 + (lane & 15) * 4;
    const float nanv = __builtin_nanf("");
#pragma unroll
    for (int i = 0; i < 9; ++i) {
      if (tr == 28 && (i < 4 || i == 8)) continue;
      float* orow = ocol + (size_t)(i < 8 ? blk * BM + i * 16 : rank * M_SH + lfrag * 16) * N_SH;
#pragma unroll
      for (int r = 0; r < 4; ++r) {
        f4 o = {acc[i][0][r], acc[i][1][r], acc[i][2][r], acc[i][3][r]};
        if (bad) o = (f4){nanv, nanv, nanv, nanv};
        asm volatile("global_store_dwordx4 %0, %1, off" EPI_BITS "\n\ts_nop 1" ::"v"(orow + (size_t)r * N_SH), "v"(o) : "memory");
      }
    }
  } else {
    __builtin_amdgcn_s_setprio(FPRIO);
    const unsigned fd = wave - 4;
    const char* gb = myws + WS_XL + (size_t)(lfrag >> 3) * NST * A_IMG;
    const char* ga = tr < 28 ? myws + WS_X16 + (size_t)(call & 1u) * X16_PAR + (size_t)blk * NST * A_IMG : gb;
    const char* fa = myws + WS_FLAGA + (size_t)(call & 1u) * FLAGA_PAR + (size_t)blk * NST * 8;
    const unsigned* fl = (const unsigned*)(myws + WS_FLAGL) + (lfrag >> 3) * NST;
    const unsigned* fw = (const unsigned*)(myws + WS_FLAGW) + cb * NST;
    unsigned* const tmo = (unsigned*)(myws + WS_TMO);
    const unsigned voff = lane * 16, voffe = EXC_OFF + lane * 2, voffl = (lfrag & 7) * 1024 + voff;
    const unsigned pl = umin(lane, 31u);
    const unsigned want = call;
    const unsigned myFull = fullAddr + fd * 4;
    bool allRaw = tr == 28;
    unsigned pbase = 0, pmask = 0, pmeta = 0;
    unsigned arrL = 0;
    unsigned freed = NSLOT;
    u4 RA[9], RB[9];
    unsigned eA0, eA1, eB0, eB1, metaA = 0, metaB = 0, tv = 0;
#define SEG(i)
#define NEXT(sf) ((sf) + 3u < (unsigned)S4 ? (sf) + 3u : (sf) < (unsigned)S4 ? S4 + fd : (sf) + 4u)
#define ARRIVED(sf) (allRaw || ((sf) - pbase < 32u && ((pmask >> ((sf) - pbase)) & 1u)))
#define POLL_LOAD(sf) __hip_atomic_load((const unsigned long long*)(fa + umin((sf) + pl, NST - 1) * 8u), __ATOMIC_RELAXED, __HIP_MEMORY_SCOPE_SYSTEM)
#define POLL_TAKE(pn, sf)                                                                    \
    {                                                                                        \
      pmask = (unsigned)__ballot((unsigned)(pn) >= want);                                    \
      pbase = (sf);                                                                          \
      pmeta = (unsigned)((pn) >> 32);                                                        \
    }
    const __amdgpu_buffer_rsrc_t ra = __builtin_amdgcn_make_buffer_rsrc((void*)ga, 0, NST * A_IMG, 0x00020000);
    const __amdgpu_buffer_rsrc_t rb = __builtin_amdgcn_make_buffer_rsrc((void*)gb, 0, NST * A_IMG, 0x00020000);
#define FETCH(R, e0, e1, meta, sf)                                                           \
    {                                                                                        \
      meta = allRaw ? META_RAW : (unsigned)__builtin_amdgcn_readlane(pmeta, ((sf) - pbase) & 31u); \
      const unsigned so = (sf) * A_IMG;                                                      \
      tv = __hip_atomic_load(tmo, __ATOMIC_RELAXED, __HIP_MEMORY_SCOPE_AGENT);               \
      const __amdgpu_buffer_rsrc_t rc = __builtin_amdgcn_make_buffer_rsrc((void*)ga, 0, meta == META_RAW ? NST * A_IMG : 0, 0x00020000); \
      _Pragma("unroll") for (int f = 0; f < 4; ++f) {                                        \
        R[4 + f] = __builtin_amdgcn_raw_buffer_load_b128(f < 3 ? ra : rc, voff + f * 1024, so + 4096, 17); \
        R[f] = __builtin_amdgcn_raw_buffer_load_b128(ra, voff + f * 1024, so, 17);           \
      }                                                                                      \
      e0 = __builtin_amdgcn_raw_buffer_load_b16(ra, voffe, so, 17);                          \
      e1 = __builtin_amdgcn_raw_buffer_load_b16(ra, voffe + 128, so, 17);                    \
      R[8] = __builtin_amdgcn_raw_buffer_load_b128(rb, voffl, so, 17);                       \
      CFENCE();                                                                              \
    }
    auto await = [&](unsigned sf) {
      while (arrL < umin(sf + 1u + LOOK, NST)) {
        const unsigned fi = umin(arrL + lane, NST - 1);
        arrL += poll_flags(fl + fi, fw + fi, want, myws, badAddr, lane);
      }
      if (!ARRIVED(sf)) {
        const unsigned long long t0 = __builtin_amdgcn_s_memrealtime();
        for (unsigned spin = 1;; ++spin) {
          const unsigned long long pn = POLL_LOAD(sf);
          POLL_TAKE(pn, sf)
          if (pmask & 1u) break;
          if ((spin & 63u) == 0u &&
              (rfl(__hip_atomic_load(tmo, __ATOMIC_RELAXED, __HIP_MEMORY_SCOPE_AGENT)) == call ||
               __builtin_amdgcn_s_memrealtime() - t0 > DEADLINE_TICKS)) {
            if (lane == 0) __hip_atomic_store(tmo, call, __ATOMIC_RELAXED, __HIP_MEMORY_SCOPE_AGENT);
            lds_st(badAddr, 1u);
            allRaw = true;
            break;
          }
        }
      }
      CFENCE();
    };
#define EMIT(R, e0, e1, meta, pub)                                                           \
    {                                                                                        \
      while (freed <= s) {                             \
        const u4 f = lds_ld4_wait(freeAddr);                                                 \
        freed = rfl(umin(umin(f.x, f.y), umin(f.z, f.w)));                                   \
        if (freed <= s) __builtin_amdgcn_s_sleep(2);                                         \
      }                                                                                      \
      SEG(1)                                                                                 \
      const unsigned slot = lds0 + (s % NSLOT) * ST;                                         \
      if (meta != META_RAW) {                                                                \
        unpack_image(R, e0, e1, slot, meta, lane);                                           \
      } else {                                                                               \
        _Pragma("unroll") for (int f = 0; f < 8; ++f)                                        \
          asm volatile("ds_write_b128 %0, %1 offset:%2" ::"v"(slot + voff), "v"(R[f]), "i"(f * 1024) : "memory"); \
      }                                                                                      \
      asm volatile("ds_write_b128 %0, %1 offset:8192" ::"v"(slot + voff), "v"(R[8]) : "memory"); \
      WAIT_L(0);                                                                             \
      SEG(2)                                                                                 \
      lds_st(myFull, pub);                                                                   \
      NG_STAMP()                                                                             \
    }
#define FEED(R, e0, e1, meta, Rn, en0, en1, metan)                                           \
    {                                                                                        \
      const unsigned sn = NEXT(s);                                                           \
      if (sn >= (unsigned)NST) {                       \
        if (rfl(tv) == call) lds_st(badAddr, 1u);                                            \
        EMIT(R, e0, e1, meta, 2u * NST)                                                      \
        break;                                                                               \
      }                                                                                      \
      if (ARRIVED(sn) && arrL >= umin(sn + 1u + LOOK, NST)) {                                \
        SEG(4)                                                                               \
        FETCH(Rn, en0, en1, metan, sn)                                                       \
        const unsigned long long pn = POLL_LOAD(sn);                                         \
        SEG(0)                                                                               \
        EMIT(R, e0, e1, meta, sn)                                                            \
        POLL_TAKE(pn, sn)                                                                    \
        SEG(3)                                                                               \
      } else {                                                                               \
        const unsigned long long pn = POLL_LOAD(sn);                                         \
        EMIT(R, e0, e1, meta, sn)                                                            \
        POLL_TAKE(pn, sn)                                                                    \
        await(sn);                                                                           \
        FETCH(Rn, en0, en1, metan, sn)                                                       \
      }                                                                                      \
      s = sn;                                                                                \
    }
#define NG_STAMP()
    unsigned s = fd < 3u ? fd : S4 + 3u;
    await(s);
    FETCH(RA, eA0, eA1, metaA, s)
    for (;;) {
      FEED(RA, eA0, eA1, metaA, RB, eB0, eB1, metaB)
      FEED(RB, eB0, eB1, metaB, RA, eA0, eA1, metaA)
    }
    STAMP(wave < 7 ? wave + 1 : 15);
  }
}

}

extern "C" void kernel_launch(int rank, int nranks, ncclComm_t comm,
                              hipStream_t stream,
                              void* const* d_in, const int* in_sizes, int n_in,
                              void* d_out, int out_size,
                              const int* in_dtypes, int out_dtype,
                              void* const* d_out_all,
                              void* const* d_ws_all, size_t ws_bytes) {
  (void)comm; (void)in_dtypes; (void)out_dtype; (void)d_out_all;
  if (nranks != NR || (unsigned)rank >= (unsigned)NR || n_in != 2 || ws_bytes < WS_END ||
      in_sizes[0] != M_SH * KD || in_sizes[1] != KD * N_TOT || out_size != NR * M_SH * N_SH) {
    fprintf(stderr, "kernel_launch: unexpected problem shape\n");
    return;
  }
  static bool attr_set[NR] = {false, false, false, false, false, false, false, false};
  if (!attr_set[rank]) {
    HIP_CHECK(hipFuncSetAttribute((const void*)fused_kernel, hipFuncAttributeMaxDynamicSharedMemorySize, LDS_BYTES));
    attr_set[rank] = true;
  }
  Params p;
  p.x = (const float*)d_in[0];
  p.w = (const float*)d_in[1];
  p.out = (float*)d_out;
  for (int i = 0; i < NR; ++i) p.ws[i] = (char*)d_ws_all[(rank + i) & (NR - 1)];
  p.rank = rank;
  p.pad = 0;
  hipLaunchKernelGGL(fused_kernel, dim3(NBLK), dim3(512), LDS_BYTES, stream, p);
  HIP_CHECK(hipGetLastError());
}
```
